```python
import numpy as np

kernel_name = "gpu_gemm_rs_m4096_k4096_n4096_e4m3e4m3_dqsilu"


ABSMAX_RTOL = 0.03

COMPUTE_DTYPE = "float32"
EPILOGUE = "dequant_silu"
IN_QUANT = None
OUT_QUANT = None
IN_FP8 = "e4m3e4m3"
IN_DTYPES = {"x": "e4m3", "w_mat": "e4m3",
             "scale_x": "float32", "scale_w": "float32"}
WORLD_SIZE = 8

def _snap_fp8(x, mant_bits, max_abs, min_normal):
    a = np.abs(x.astype(np.float64))
    _m, _e = np.frexp(a)
    q = np.where(a >= min_normal, np.ldexp(1.0, _e - 1 - mant_bits),
                 min_normal * 2.0 ** -mant_bits)
    snapped = np.minimum(np.round(a / q) * q, max_abs)
    return (np.sign(x) * snapped).astype(np.float32)

def setup_inputs():

    x = _snap_fp8(np.random.randn(4096, 4096).astype(np.float32), 3, 448.0, 2.0 ** -6)
    w_mat = _snap_fp8(np.random.randn(4096, 4096).astype(np.float32), 3, 448.0, 2.0 ** -6)
    scale_x = (0.02 * (1.0 + np.random.rand(1))).astype(np.float32)
    scale_w = (0.02 * (1.0 + np.random.rand(1))).astype(np.float32)
    return {"x": x, "w_mat": w_mat, "scale_x": scale_x, "scale_w": scale_w}

def reference(x, w_mat, scale_x, scale_w):
    acc = (x @ w_mat).astype(np.float32)
    y = (acc * (scale_x * scale_w)).astype(np.float32)
    return (y / (1.0 + np.exp(-np.clip(y, -60.0, 60.0)))).astype(np.float32)
```

```cpp
#include <hip/hip_runtime.h>
#include <rccl/rccl.h>
#include <cstdio>

#define HIP_CHECK(expr) do { hipError_t _e = (expr); if (_e != hipSuccess) { \
    fprintf(stderr, "HIP error %s at %s:%d\n", hipGetErrorString(_e), __FILE__, __LINE__); \
    return; } } while (0)

namespace {

constexpr int NR = 8;
constexpr int MS = 512;
constexpr int KS = 512;
constexpr int NN = 4096;
constexpr int NWG = 256;
constexpr int ROUNDS = KS / 16;
constexpr int RING = 16;
#define WIN 8

constexpr size_t WST_BYTES = (size_t)KS * NN;
constexpr size_t XST_BYTES = (size_t)MS * KS * NR;
constexpr size_t SLOT_BYTES = WST_BYTES + XST_BYTES;
constexpr size_t FW_OFF = 2 * SLOT_BYTES;
constexpr size_t FX_OFF = FW_OFF + 64 * 256;
constexpr size_t CNT_OFF = FX_OFF + 16 * 128;
constexpr size_t TMO_OFF = CNT_OFF + NWG * 128;
constexpr size_t WS_NEED = TMO_OFF + 128;
constexpr unsigned long long DEADLINE_TICKS = 50000000ull;

typedef unsigned int u32x4 __attribute__((ext_vector_type(4)));
typedef int i32x8 __attribute__((ext_vector_type(8)));
typedef int i32x4 __attribute__((ext_vector_type(4)));
typedef float f32x4 __attribute__((ext_vector_type(4)));

struct Params {
    const unsigned char* x;
    const unsigned char* w;
    const float* sx;
    const float* sw;
    float* out;
    unsigned char* ws[NR];
    int rank;
};

__device__ __forceinline__ void store16_sys(void* p, u32x4 v) {
    asm volatile("global_store_dwordx4 %0, %1, off sc0 sc1\n\ts_nop 1" :: "v"(p), "v"(v) : "memory");
}
__device__ __forceinline__ void lds_st(unsigned addr, unsigned v) {
    asm volatile("ds_write_b32 %0, %1" :: "v"(addr), "v"(v) : "memory");
}
__device__ __forceinline__ unsigned lds_ld(unsigned addr) {
    unsigned v;
    asm volatile("ds_read_b32 %0, %1\n\ts_waitcnt lgkmcnt(0)" : "=v"(v) : "v"(addr) : "memory");
    return v;
}
__device__ __forceinline__ void glds16(const unsigned char* sbase, unsigned voff, unsigned lds_dst) {
    unsigned keep;
    asm volatile("s_mov_b32 %0, m0\n\ts_mov_b32 m0, %3\n\ts_nop 0\n\tglobal_load_lds_dwordx4 %1, %2\n\ts_mov_b32 m0, %0"
                 : "=&s"(keep) : "v"(voff), "s"(sbase), "s"(lds_dst) : "memory");
}
template <int N> __device__ __forceinline__ void wait_vm() { asm volatile("s_waitcnt vmcnt(%0)" :: "i"(N) : "memory"); }

__device__ __forceinline__ void wait_flags(const unsigned* flag, bool active, unsigned call, unsigned* tmo) {
    const unsigned long long t0 = __builtin_amdgcn_s_memrealtime();
    for (;;) {
        unsigned v = call;
        if (active) v = __hip_atomic_load(flag, __ATOMIC_RELAXED, __HIP_MEMORY_SCOPE_SYSTEM);
        if (__all(v >= call)) break;
        if (__hip_atomic_load(tmo, __ATOMIC_RELAXED, __HIP_MEMORY_SCOPE_AGENT) == call
            || __builtin_amdgcn_s_memrealtime() - t0 > DEADLINE_TICKS) {
            __hip_atomic_store(tmo, call, __ATOMIC_RELAXED, __HIP_MEMORY_SCOPE_AGENT);
            break;
        }
        __builtin_amdgcn_s_sleep(2);
    }
    __builtin_amdgcn_fence(__ATOMIC_ACQUIRE, "");
}

__global__ void __launch_bounds__(320, 1) gemm_gather_silu(Params p) {
    __shared__ __attribute__((aligned(16))) unsigned char smem[RING * 8192 + 64];
    const int tid = threadIdx.x, lane = tid & 63;
    const int wave = __builtin_amdgcn_readfirstlane(tid >> 6);
    const int b = blockIdx.x;
    const int g = (b & 7) * 8 + (b >> 5);
    const int mt = (b >> 3) & 3;
    const int rank = p.rank;
    unsigned char* const myws = p.ws[rank];
    unsigned* const tmo = (unsigned*)(myws + TMO_OFF);
    unsigned* const cnt = (unsigned*)(myws + CNT_OFF + (size_t)b * 128) + wave;
    const unsigned lds0 = (unsigned)(size_t)(__attribute__((address_space(3))) unsigned char*)smem;
    const unsigned CTL = lds0 + RING * 8192;

    const unsigned cnt_old = __hip_atomic_load(cnt, __ATOMIC_RELAXED, __HIP_MEMORY_SCOPE_AGENT);
    if (tid < 8) lds_st(CTL + tid * 4, 0u);
    asm volatile("s_waitcnt lgkmcnt(0)" ::: "memory");
    __builtin_amdgcn_s_barrier();
    unsigned call;
    size_t slot;
#define GET_CALL() do { call = __builtin_amdgcn_readfirstlane(cnt_old) + 1u; slot = (size_t)(call & 1u) * SLOT_BYTES; \
        if (lane == 0) __hip_atomic_store(cnt, call, __ATOMIC_RELAXED, __HIP_MEMORY_SCOPE_AGENT); } while (0)
    unsigned char* const peer_l = p.ws[lane & 7];

    if (wave < 2) {
        const int cg = lane & 15, kq = lane >> 4;
        const int r = mt * 8 + wave * 4 + kq;
        const unsigned char* src = p.w + (size_t)(r * 16) * NN + g * 64 + cg * 4;
        unsigned d[16];
#pragma unroll
        for (int i = 0; i < 16; ++i) d[i] = *(const unsigned*)(src + (size_t)i * NN);
        GET_CALL();
        u32x4 o[4];
#pragma unroll
        for (int q = 0; q < 4; ++q) {
            const unsigned t0 = __builtin_amdgcn_perm(d[4 * q + 1], d[4 * q], 0x05010400u);
            const unsigned t1 = __builtin_amdgcn_perm(d[4 * q + 1], d[4 * q], 0x07030602u);
            const unsigned t2 = __builtin_amdgcn_perm(d[4 * q + 3], d[4 * q + 2], 0x05010400u);
            const unsigned t3 = __builtin_amdgcn_perm(d[4 * q + 3], d[4 * q + 2], 0x07030602u);
            o[0][q] = __builtin_amdgcn_perm(t2, t0, 0x05040100u);
            o[1][q] = __builtin_amdgcn_perm(t2, t0, 0x07060302u);
            o[2][q] = __builtin_amdgcn_perm(t3, t1, 0x05040100u);
            o[3][q] = __builtin_amdgcn_perm(t3, t1, 0x07060302u);
        }
        unsigned char* dst = myws + slot + (size_t)g * 32768 + r * 1024 + cg * 16;
#pragma unroll
        for (int j = 0; j < 4; ++j) store16_sys(dst + j * 256, o[j]);
        asm volatile("s_waitcnt vmcnt(0)" ::: "memory");
        __builtin_amdgcn_fence(__ATOMIC_RELEASE, "");
        asm volatile("s_waitcnt vmcnt(0)" ::: "memory");
        if (lane < NR)
            __hip_atomic_store((unsigned*)(peer_l + FW_OFF + g * 256) + rank * 8 + mt * 2 + wave, call,
                               __ATOMIC_RELAXED, __HIP_MEMORY_SCOPE_SYSTEM);
    } else if (wave < 4) {
        const int half = wave - 2;
        const unsigned dest = (unsigned)(rank + (b & 7)) & 7u;
        const int rt = b >> 3;
        const int row = lane & 15, rr = lane >> 4;
        const unsigned char* src = p.x + (size_t)(dest * MS + rt * 16 + row) * KS + (half * 16 + rr) * 16;
        u32x4 v[4];
#pragma unroll
        for (int i = 0; i < 4; ++i) v[i] = *(const u32x4*)(src + i * 64);
        GET_CALL();
        unsigned char* const dws = p.ws[dest];
        unsigned char* dst = dws + slot + WST_BYTES + (size_t)rt * 65536 + (half * 16 + rr) * 2048
                             + (rank & 1) * 1024 + (rank >> 1) * 256 + row * 16;
#pragma unroll
        for (int i = 0; i < 4; ++i) store16_sys(dst + i * 8192, v[i]);
        asm volatile("s_waitcnt vmcnt(0)" ::: "memory");
        __builtin_amdgcn_fence(__ATOMIC_RELEASE, "");
        asm volatile("s_waitcnt vmcnt(0)" ::: "memory");
        if (lane == 0)
            __hip_atomic_store((unsigned*)(dws + FX_OFF + (rt >> 1) * 128) + ((rt & 1) * 8 + rank) * 2 + half, call,
                               __ATOMIC_RELAXED, __HIP_MEMORY_SCOPE_SYSTEM);
    } else {
        __builtin_amdgcn_s_setprio(3);
        GET_CALL();
        const unsigned char* base[NR];
        unsigned ldst[NR];
#pragma unroll
        for (int i = 0; i < NR; ++i) {
            const unsigned s = (unsigned)(rank + g + i) & 7u;
            base[i] = p.ws[s] + slot + (size_t)g * 32768;
            ldst[i] = lds0 + s * 1024;
        }
        wait_flags((const unsigned*)(myws + FW_OFF + g * 256) + lane, true, call, tmo);
        asm volatile("s_waitcnt vmcnt(0)" ::: "memory");
        const bool dead = __builtin_amdgcn_readfirstlane(
            __hip_atomic_load(tmo, __ATOMIC_RELAXED, __HIP_MEMORY_SCOPE_AGENT)) == call;
        if (!dead) {
            unsigned voff = lane * 16;
#pragma unroll 1
            for (int r = 0; r < ROUNDS; ++r) {
                if (r >= RING) {
                    for (;;) {
                        const unsigned dn = lds_ld(CTL + 4 + (lane & 3) * 4);
                        if (__all(dn + RING > (unsigned)r)) break;
                        __builtin_amdgcn_s_sleep(1);
                    }
                }
                const unsigned so = (unsigned)(r % RING) * 8192;
#pragma unroll
                for (int i = 0; i < NR; ++i) {
                    glds16(base[i], voff, ldst[i] + so);
                    wait_vm<WIN - 1>();
                    if (((i + 8 * 8 - (WIN - 1)) & 7) == 7) {
                        const int landed = (8 * r + i - (WIN - 1) + 1) / 8;
                        if (8 * r + i - (WIN - 1) >= 7) lds_st(CTL, (unsigned)landed);
                    }
                }
                voff += 1024;
            }
            wait_vm<0>();
        }
        lds_st(CTL, ROUNDS);
        return;
    }

    wait_flags((const unsigned*)(myws + FX_OFF + (mt * 4 + wave) * 128) + (lane & 31), true, call, tmo);
    const unsigned char* abase = myws + slot + WST_BYTES + (size_t)(mt * 8 + wave * 2) * 65536 + lane * 16;
    constexpr int PF = 4;
    i32x4 a[PF][2][2];
#pragma unroll
    for (int u = 0; u < PF; ++u)
#pragma unroll
        for (int t = 0; t < 2; ++t)
#pragma unroll
            for (int h = 0; h < 2; ++h)
                a[u][t][h] = *(const i32x4*)(abase + t * 65536 + u * 2048 + h * 1024);
    f32x4 acc[2][4];
#pragma unroll
    for (int t = 0; t < 2; ++t)
#pragma unroll
        for (int j = 0; j < 4; ++j) acc[t][j] = f32x4{0.f, 0.f, 0.f, 0.f};
    const unsigned char* bl = smem + (lane >> 4) * 2048 + (lane & 15) * 16;
    unsigned landed = 0;
#pragma unroll 1
    for (int r0 = 0; r0 < ROUNDS; r0 += PF) {
#pragma unroll
        for (int u = 0; u < PF; ++u) {
            const int r = r0 + u;
            while (landed <= (unsigned)r) {
                landed = __builtin_amdgcn_readfirstlane(lds_ld(CTL));
                if (landed <= (unsigned)r) __builtin_amdgcn_s_sleep(1);
            }
            const unsigned char* bs = bl + (r % RING) * 8192;
            i32x8 bf[4];
#pragma unroll
            for (int j = 0; j < 4; ++j) {
                const i32x4 lo = *(const i32x4*)(bs + j * 256);
                const i32x4 hi = *(const i32x4*)(bs + 1024 + j * 256);
                bf[j] = __builtin_shufflevector(lo, hi, 0, 1, 2, 3, 4, 5, 6, 7);
            }
            lds_st(CTL + 4 + wave * 4, (unsigned)r + 1u);
#pragma unroll
            for (int t = 0; t < 2; ++t) {
                const i32x8 af = __builtin_shufflevector(a[u][t][0], a[u][t][1], 0, 1, 2, 3, 4, 5, 6, 7);
#pragma unroll
                for (int j = 0; j < 4; ++j)
                    acc[t][j] = __builtin_amdgcn_mfma_scale_f32_16x16x128_f8f6f4(af, bf[j], acc[t][j], 0, 0,
                                                                                 0, 0x7F7F7F7F, 0, 0x7F7F7F7F);
            }
            const int rn = (r + PF < ROUNDS) ? r + PF : r;
#pragma unroll
            for (int t = 0; t < 2; ++t)
#pragma unroll
                for (int h = 0; h < 2; ++h)
                    a[u][t][h] = *(const i32x4*)(abase + t * 65536 + rn * 2048 + h * 1024);
        }
    }
    const float sc = p.sx[0] * p.sw[0];
    const bool dead = __hip_atomic_load(tmo, __ATOMIC_RELAXED, __HIP_MEMORY_SCOPE_AGENT) == call;
    float* orow = p.out + (size_t)(mt * 128 + wave * 32 + (lane >> 4) * 4) * NN + g * 64 + (lane & 15) * 4;
#pragma unroll
    for (int t = 0; t < 2; ++t)
#pragma unroll
        for (int i = 0; i < 4; ++i) {
            f32x4 o;
#pragma unroll
            for (int j = 0; j < 4; ++j) {
                const float y = acc[t][j][i] * sc;
                const float e = __expf(-fminf(fmaxf(y, -60.f), 60.f));
                o[j] = dead ? __builtin_nanf("") : y / (1.f + e);
            }
            *(f32x4*)(orow + (size_t)(t * 16 + i) * NN) = o;
        }
}

}

extern "C" void kernel_launch(int rank, int nranks, ncclComm_t comm, hipStream_t stream,
                              void* const* d_in, const int* in_sizes, int n_in,
                              void* d_out, int out_size, const int* in_dtypes, int out_dtype,
                              void* const* d_out_all, void* const* d_ws_all, size_t ws_bytes) {
    (void)comm; (void)in_sizes; (void)n_in; (void)out_size; (void)in_dtypes; (void)out_dtype; (void)d_out_all;
    if (nranks != NR || ws_bytes < WS_NEED || (unsigned)rank >= (unsigned)NR) {
        fprintf(stderr, "unexpected configuration\n");
        return;
    }
    Params p;
    p.x = (const unsigned char*)d_in[0];
    p.w = (const unsigned char*)d_in[1];
    p.sx = (const float*)d_in[2];
    p.sw = (const float*)d_in[3];
    p.out = (float*)d_out;
    for (int i = 0; i < NR; ++i) p.ws[i] = (unsigned char*)d_ws_all[i];
    p.rank = rank;
    gemm_gather_silu<<<NWG, 320, 0, stream>>>(p);
    HIP_CHECK(hipGetLastError());
}
```

```cpp
#include <hip/hip_runtime.h>
#include <rccl/rccl.h>
#include <cstdio>

#define HIP_CHECK(expr) do { hipError_t _e = (expr); if (_e != hipSuccess) { \
    fprintf(stderr, "HIP error %s at %s:%d\n", hipGetErrorString(_e), __FILE__, __LINE__); \
    return; } } while (0)

namespace {

constexpr int NR = 8;
constexpr int MS = 512;
constexpr int KS = 512;
constexpr int NN = 4096;
constexpr int NWG = 256;
constexpr int ROUNDS = KS / 16;
constexpr int RING = 16;
#define OST 2
#define SDELAY 8
#define XR0 3
#define XRSPAN 20
#define PACE 107
#define TOFF 60
#define TREFMODE 0
#define XGATE 3
#define FXSCOPE __HIP_MEMORY_SCOPE_SYSTEM
#define STHOLD 0
#define SMOOTH 0
#define CATCH 4
#define LAG 60
#define ENDL 0
constexpr int WIN = 8;

constexpr size_t WST_BYTES = (size_t)KS * NN;
constexpr size_t XSRC_BYTES = (size_t)NR * MS * KS;
constexpr size_t XST_BYTES = (size_t)MS * KS * NR;
constexpr size_t XSRC_OFF = WST_BYTES;
constexpr size_t XST_OFF = WST_BYTES + XSRC_BYTES;
constexpr size_t SLOT_BYTES = WST_BYTES + XSRC_BYTES + XST_BYTES;
constexpr size_t FW_OFF = 2 * SLOT_BYTES;
constexpr size_t FS_OFF = FW_OFF + 64 * 256;
constexpr size_t FX_OFF = FS_OFF + 64 * 128;
constexpr size_t CNT_OFF = FX_OFF + 16 * 128;
constexpr size_t TMO_OFF = CNT_OFF + NWG * 128;
constexpr size_t INV_OFF = TMO_OFF + 128;
constexpr size_t TREF_OFF = INV_OFF + 16 * 256;
constexpr size_t WS_NEED = TREF_OFF + 8 * 256;
constexpr unsigned long long DEADLINE_TICKS = 50000000ull;

#define STAMP(i) do {} while (0)
typedef unsigned int u32x4 __attribute__((ext_vector_type(4)));
typedef int i32x8 __attribute__((ext_vector_type(8)));
typedef int i32x4 __attribute__((ext_vector_type(4)));
typedef float f32x4 __attribute__((ext_vector_type(4)));

struct Params {
    const unsigned char* x;
    const unsigned char* w;
    const float* sx;
    const float* sw;
    float* out;
    unsigned char* myws;
    unsigned char* ws[NR];
    int rank;
};

__device__ __forceinline__ void store16_sys(void* p, u32x4 v) {
    asm volatile("global_store_dwordx4 %0, %1, off sc0 sc1\n\ts_nop 1" :: "v"(p), "v"(v) : "memory");
}
__device__ __forceinline__ void lds_st(unsigned addr, unsigned v) {
    asm volatile("ds_write_b32 %0, %1" :: "v"(addr), "v"(v) : "memory");
}
__device__ __forceinline__ unsigned lds_ld(unsigned addr) {
    unsigned v;
    asm volatile("ds_read_b32 %0, %1\n\ts_waitcnt lgkmcnt(0)" : "=v"(v) : "v"(addr) : "memory");
    return v;
}
__device__ __forceinline__ unsigned lds_wait_nz(unsigned addr) {
    unsigned v;
    while ((v = __builtin_amdgcn_readfirstlane(lds_ld(addr))) == 0u) __builtin_amdgcn_s_sleep(1);
    return v;
}
__device__ __forceinline__ void glds16(const unsigned char* sbase, unsigned voff, unsigned lds_dst) {
    unsigned keep;
    asm volatile("s_mov_b32 %0, m0\n\ts_mov_b32 m0, %3\n\ts_nop 0\n\tglobal_load_lds_dwordx4 %1, %2\n\ts_mov_b32 m0, %0"
                 : "=&s"(keep) : "v"(voff), "s"(sbase), "s"(lds_dst) : "memory");
}
template <int N> __device__ __forceinline__ void wait_vm() { asm volatile("s_waitcnt vmcnt(%0)" :: "i"(N) : "memory"); }

template <int SLEEP, int SCOPE = __HIP_MEMORY_SCOPE_SYSTEM>
__device__ __forceinline__ bool wait_flags(const unsigned* flag, unsigned call, unsigned* tmo) {
    const unsigned long long t0 = __builtin_amdgcn_s_memrealtime();
    for (unsigned spin = 1;; ++spin) {
        const unsigned v = __hip_atomic_load(flag, __ATOMIC_RELAXED, SCOPE);
        if (__all(v >= call)) break;
        if ((spin & 63u) == 0u) {
            if (__hip_atomic_load(tmo, __ATOMIC_RELAXED, __HIP_MEMORY_SCOPE_AGENT) == call
                || __builtin_amdgcn_s_memrealtime() - t0 > DEADLINE_TICKS) {
                __hip_atomic_store(tmo, call, __ATOMIC_RELAXED, __HIP_MEMORY_SCOPE_AGENT);
                return false;
            }
        }
        __builtin_amdgcn_s_sleep(SLEEP);
    }
    asm volatile("" ::: "memory");
    return true;
}

__global__ void __launch_bounds__(384, 1) gemm_gather_silu(Params p) {
    __shared__ __attribute__((aligned(16))) unsigned char smem[RING * 8192 + 8192 + 64];
    const int tid = threadIdx.x, lane = tid & 63;
    const int wave = __builtin_amdgcn_readfirstlane(tid >> 6);
    const int b = blockIdx.x;
    const int g = (b & 7) * 8 + (b >> 5);
    const int mt = (b >> 3) & 3;
    const int xo = mt * 8 + (b >> 5);
    const int xrq = xo >> 2, xrt = (xo & 3) * 8 + (b & 7);
    const int rank = p.rank;
    unsigned char* const myws = p.myws;
    unsigned* const tmo = (unsigned*)(myws + TMO_OFF);
    unsigned* const cnt = (unsigned*)(myws + CNT_OFF + (size_t)b * 128) + wave;
    const unsigned lds0 = (unsigned)(size_t)(__attribute__((address_space(3))) unsigned char*)smem;
    const unsigned CTL = lds0 + RING * 8192 + 8192;

    const unsigned t_start = (unsigned)__builtin_amdgcn_s_memrealtime();
    if (wave == 0) STAMP(0);
    const unsigned cnt_old = __hip_atomic_load(cnt, __ATOMIC_RELAXED, __HIP_MEMORY_SCOPE_AGENT);
    if (tid < 16) lds_st(CTL + tid * 4, 0u);
    asm volatile("s_waitcnt lgkmcnt(0)" ::: "memory");
    __builtin_amdgcn_s_barrier();
    unsigned call;
    size_t slot;
#define GET_CALL() do { call = __builtin_amdgcn_readfirstlane(cnt_old) + 1u; slot = (size_t)(call & 1u) * SLOT_BYTES; \
        if (lane == 0) __hip_atomic_store(cnt, call, __ATOMIC_RELAXED, __HIP_MEMORY_SCOPE_AGENT); } while (0)

    if (wave < 2) {
        unsigned char* peer_l = p.ws[lane & 7];
        const int kb = mt * 2 + wave;
        if (SDELAY && kb != 0) __builtin_amdgcn_s_sleep(SDELAY);
        const int cg = lane & 15, kq = lane >> 4;
        const int r = kb * 4 + kq;
        const unsigned char* src = p.w + (size_t)(r * 16) * NN + g * 64 + cg * 4;
        unsigned d[16];
#pragma unroll
        for (int i = 0; i < 16; ++i) d[i] = *(const unsigned*)(src + (size_t)i * NN);
        asm volatile("" : "+v"(peer_l));
        GET_CALL();
        u32x4 o[4];
#pragma unroll
        for (int q = 0; q < 4; ++q) {
            const unsigned t0 = __builtin_amdgcn_perm(d[4 * q + 1], d[4 * q], 0x05010400u);
            const unsigned t1 = __builtin_amdgcn_perm(d[4 * q + 1], d[4 * q], 0x07030602u);
            const unsigned t2 = __builtin_amdgcn_perm(d[4 * q + 3], d[4 * q + 2], 0x05010400u);
            const unsigned t3 = __builtin_amdgcn_perm(d[4 * q + 3], d[4 * q + 2], 0x07030602u);
            o[0][q] = __builtin_amdgcn_perm(t2, t0, 0x05040100u);
            o[1][q] = __builtin_amdgcn_perm(t2, t0, 0x07060302u);
            o[2][q] = __builtin_amdgcn_perm(t3, t1, 0x05040100u);
            o[3][q] = __builtin_amdgcn_perm(t3, t1, 0x07060302u);
        }
        unsigned char* dst = myws + slot + (size_t)g * 32768 + r * 1024 + cg * 16;
        if (STHOLD && kb != 0) while ((unsigned)__builtin_amdgcn_s_memrealtime() - t_start < STHOLD) __builtin_amdgcn_s_sleep(1);
#pragma unroll
        for (int j = 0; j < 4; ++j) store16_sys(dst + j * 256, o[j]);
        if (kb == 0) STAMP(1);
        asm volatile("s_waitcnt vmcnt(0)" ::: "memory");
        if (kb == 0) STAMP(2);
        if (kb == 7) STAMP(13);
        if (lane < NR)
            __hip_atomic_store((unsigned*)(peer_l + FW_OFF + g * 256) + kb * 8 + rank, call,
                               __ATOMIC_RELAXED, __HIP_MEMORY_SCOPE_SYSTEM);
    } else if (wave < 4) {
        if (SDELAY) __builtin_amdgcn_s_sleep(SDELAY);
        const int half = wave - 2;
        const unsigned dest = (unsigned)(rank + b) & 7u;
        const int rt = b >> 3;
        const int row = lane & 15, rr = lane >> 4;
        const unsigned char* src = p.x + (size_t)(dest * MS + rt * 16 + row) * KS + (half * 16 + rr) * 16;
        u32x4 v[4];
#pragma unroll
        for (int i = 0; i < 4; ++i) v[i] = *(const u32x4*)(src + i * 64);
        unsigned char* const dws = p.ws[dest];
        GET_CALL();
        unsigned char* dst = myws + slot + XSRC_OFF + (size_t)dest * (MS * KS) + (rt * 8 + half * 4) * 1024 + lane * 16;
        if (STHOLD) while ((unsigned)__builtin_amdgcn_s_memrealtime() - t_start < STHOLD) __builtin_amdgcn_s_sleep(1);
#pragma unroll
        for (int i = 0; i < 4; ++i) store16_sys(dst + i * 1024, v[i]);
        if (wave == 2) STAMP(4);
        asm volatile("s_waitcnt vmcnt(0)" ::: "memory");
        if (wave == 2) STAMP(5);
        if (lane == 0)
            __hip_atomic_store((unsigned*)(dws + FS_OFF + (rt * 2 + half) * 128) + rank, call,
                               __ATOMIC_RELAXED, __HIP_MEMORY_SCOPE_SYSTEM);
    } else if (wave == 5) {
        GET_CALL();
        lds_wait_nz(CTL + 36);
        bool ok = true;
        {
            const unsigned* f1 = (const unsigned*)(myws + FW_OFF + g * 256) + lane;
            const unsigned* f2 = (const unsigned*)(myws + FS_OFF + (xrt * 2 + (xrq >> 2)) * 128) + (lane & 7);
            const unsigned long long t0 = __builtin_amdgcn_s_memrealtime();
            for (unsigned spin = 1;; ++spin) {
                __builtin_amdgcn_s_sleep(16);
                unsigned v1, v2;
                asm volatile("global_load_dword %0, %2, off sc0 sc1\n\tglobal_load_dword %1, %3, off sc0 sc1\n\ts_waitcnt vmcnt(0)"
                             : "=&v"(v1), "=&v"(v2) : "v"(f1), "v"(f2) : "memory");
                if (__all(v1 >= call && v2 >= call)) break;
                if ((spin & 63u) == 0u) {
                    if (__hip_atomic_load(tmo, __ATOMIC_RELAXED, __HIP_MEMORY_SCOPE_AGENT) == call
                        || __builtin_amdgcn_s_memrealtime() - t0 > DEADLINE_TICKS) {
                        __hip_atomic_store(tmo, call, __ATOMIC_RELAXED, __HIP_MEMORY_SCOPE_AGENT);
                        ok = false;
                        break;
                    }
                }
            }
        }
        lds_st(CTL + 20, ok ? 1u : 2u);
        if (TREFMODE) {
            const unsigned long long tr = __hip_atomic_load((unsigned long long*)(myws + TREF_OFF + (b & 7) * 256), __ATOMIC_RELAXED, __HIP_MEMORY_SCOPE_AGENT);
            lds_st(CTL + 32, (unsigned)(tr >> 32) == call ? ~(unsigned)tr : 0u);
        }
        STAMP(12);
        unsigned xl;
        while ((xl = __builtin_amdgcn_readfirstlane(lds_ld(CTL + 24))) == 0u) __builtin_amdgcn_s_sleep(8);
        if (xl == 1u) {
            unsigned char* dst = myws + slot + XST_OFF + (size_t)xrt * 65536 + (xrq * 4 + (lane >> 4)) * 2048 + (lane & 15) * 16;
#pragma unroll
            for (int s = 0; s < NR; ++s) {
                const u32x4 v = *(const u32x4*)(smem + RING * 8192 + s * 1024 + lane * 16);
                store16_sys(dst + (s & 1) * 1024 + (s >> 1) * 256, v);
            }
            asm volatile("s_waitcnt vmcnt(0)" ::: "memory");
            if (lane == 0)
                __hip_atomic_store((unsigned*)(myws + FX_OFF + (xrt >> 1) * 128) + xrq * 2 + (xrt & 1), call,
                                   __ATOMIC_RELAXED, __HIP_MEMORY_SCOPE_SYSTEM);
        }
        STAMP(3);
        return;
    } else {
        __builtin_amdgcn_s_setprio(3);
        const unsigned xcc = (unsigned)__builtin_amdgcn_s_getreg((3 << 11) | 20) & 0xFu;
        unsigned* const iv = (unsigned*)(myws + INV_OFF + xcc * 256);
        bool inv_done = false;
        if (b < 8) {
            asm volatile("buffer_inv sc0 sc1\n\ts_waitcnt vmcnt(0)" ::: "memory");
            inv_done = true;
        }
        GET_CALL();
        if (b < 8 && lane == 0) __hip_atomic_store(iv, call, __ATOMIC_RELAXED, __HIP_MEMORY_SCOPE_AGENT);
        const unsigned char* base[NR];
        unsigned ldst[NR];
#pragma unroll
        for (int i = 0; i < NR; ++i) {
            const unsigned s = (unsigned)(rank + g + i) & 7u;
            base[i] = p.ws[s] + slot;
            ldst[i] = lds0 + s * 1024;
        }
        STAMP(6);
        bool ok = true;
        {
            const unsigned* fw = lane < NR ? (const unsigned*)(myws + FW_OFF + g * 256) + lane : iv;
            const unsigned long long t0 = __builtin_amdgcn_s_memrealtime();
            for (unsigned spin = 1;; ++spin) {
                const unsigned v = __hip_atomic_load(fw, __ATOMIC_RELAXED, __HIP_MEMORY_SCOPE_SYSTEM);
                const unsigned long long rdy = __ballot(v >= call);
                inv_done = inv_done || (rdy >> 63);
                if ((rdy & 0xFFull) == 0xFFull && inv_done) break;
                if (inv_done) fw = (const unsigned*)(myws + FW_OFF + g * 256) + (lane & 7);
                if ((spin & 31u) == 0u) {
                    if (!inv_done) {
                        asm volatile("buffer_inv sc0 sc1\n\ts_waitcnt vmcnt(0)" ::: "memory");
                        inv_done = true;
                    }
                    if (__hip_atomic_load(tmo, __ATOMIC_RELAXED, __HIP_MEMORY_SCOPE_AGENT) == call
                        || __builtin_amdgcn_s_memrealtime() - t0 > DEADLINE_TICKS) {
                        __hip_atomic_store(tmo, call, __ATOMIC_RELAXED, __HIP_MEMORY_SCOPE_AGENT);
                        ok = false;
                        break;
                    }
                }
                __builtin_amdgcn_s_sleep(1);
            }
        }
        const unsigned long long t7 = __builtin_amdgcn_s_memrealtime();
        unsigned tref = (unsigned)t7;
        lds_st(CTL + 36, tref | 1u);
        const int xr = XR0 + xo * XRSPAN / 32;
        const int chk = xr < 4 ? xr : 4;
        unsigned voff = g * 32768 + lane * 16;
        unsigned pub_addr = CTL + 28, pub_val = 0;
#pragma unroll 1
        for (int r = 0; r < ROUNDS - ENDL && ok; ++r) {
            if (r == chk) ok = lds_wait_nz(CTL + 20) == 1u;
            if (TREFMODE && r == 8) {
                const unsigned tr = __builtin_amdgcn_readfirstlane(lds_ld(CTL + 32));
                if (tr != 0u && tref + 1u - tr < 1000u) tref = tr + TOFF;
            }
            unsigned sched = 0, sstep = 0;
            bool behind = false;
            if (PACE) {
                int xi = r - XR0; xi = xi < 0 ? 0 : (xi > XRSPAN ? XRSPAN : xi);
                sched = (unsigned)(PACE * (r * XRSPAN + 4 * xi)) / XRSPAN;
                sstep = (r >= XR0 && r < XR0 + XRSPAN) ? (unsigned)(PACE * (XRSPAN + 4)) / (XRSPAN * 8) : PACE / 8;
                const unsigned now = (unsigned)__builtin_amdgcn_s_memrealtime() - tref;
                behind = now > sched + LAG;
                if (now < sched) {
                    while ((unsigned)__builtin_amdgcn_s_memrealtime() - tref < sched) __builtin_amdgcn_s_sleep(1);
                }
            }
            if (r == xr && ok) {
                const unsigned xoff = (unsigned)XSRC_OFF + rank * (MS * KS) + (xrt * 8 + xrq) * 1024 + lane * 16;
#pragma unroll
                for (int i = 0; i < NR; ++i) {
                    glds16(base[i], xoff, ldst[i] + RING * 8192);
                    wait_vm<WIN - 1>();
                    if (i == NR - 2) lds_st(pub_addr, pub_val);
                }
                pub_addr = CTL + 24; pub_val = 1u;
            }
            if (r >= RING) {
                for (;;) {
                    const unsigned dn = lds_ld(CTL + 4 + (lane & 3) * 4);
                    if (__all(dn + RING > (unsigned)r)) break;
                    wait_vm<0>();
                    lds_st(pub_addr, pub_val);
                    __builtin_amdgcn_s_sleep(1);
                }
            }
            if (!ok) break;
            const unsigned so = (unsigned)(r % RING) * 8192;
            if (CATCH && behind) {
#pragma unroll
                for (int i = 0; i < NR; ++i) {
                    glds16(base[i], voff, ldst[i] + so);
                    wait_vm<WIN - 1 + CATCH>();
                }
                wait_vm<8>();
                lds_st(pub_addr, pub_val);
            } else {
#pragma unroll
                for (int i = 0; i < NR; ++i) {
                    if (SMOOTH && i > 0) {
                        while ((unsigned)__builtin_amdgcn_s_memrealtime() - tref < sched + i * sstep) __builtin_amdgcn_s_sleep(1);
                    }
                    glds16(base[i], voff, ldst[i] + so);
                    wait_vm<WIN - 1>();
                    if (i == NR - 2) lds_st(pub_addr, pub_val);
                }
            }
            pub_addr = CTL; pub_val = (unsigned)r + 1u;
            voff += 1024;
        }
        if (ENDL && ok) {
            for (;;) {
                const unsigned dn = lds_ld(CTL + 4 + (lane & 3) * 4);
                if (__all(dn + RING > (unsigned)(ROUNDS - 1))) break;
                wait_vm<0>();
                lds_st(pub_addr, pub_val);
                __builtin_amdgcn_s_sleep(1);
            }
#pragma unroll
            for (int e = 0; e < ENDL; ++e) {
                const unsigned so = (unsigned)((ROUNDS - ENDL + e) % RING) * 8192;
#pragma unroll
                for (int i = 0; i < NR; ++i) glds16(base[i], voff, ldst[i] + so);
                voff += 1024;
            }
            wait_vm<ENDL * 8>(); lds_st(pub_addr, pub_val);
            if (ENDL > 3) { wait_vm<24>(); lds_st(CTL, ROUNDS - 3); }
            if (ENDL > 2) { wait_vm<16>(); lds_st(CTL, ROUNDS - 2); }
            if (ENDL > 1) { wait_vm<8>(); lds_st(CTL, ROUNDS - 1); }
        }
        wait_vm<0>();
        if (!ok) lds_st(CTL + 24, 2u);
        lds_st(CTL, ok ? (unsigned)ROUNDS : 0x80000000u);
        STAMP(8);
        return;
    }

    if (TREFMODE && wave == 0 && mt == 0) {
        const unsigned ts = lds_wait_nz(CTL + 36);
        if (lane == 0) {
            const unsigned long long tv = ((unsigned long long)call << 32) | (unsigned)~ts;
            __hip_atomic_fetch_max((unsigned long long*)(myws + TREF_OFF + (b & 7) * 256), tv, __ATOMIC_RELAXED, __HIP_MEMORY_SCOPE_AGENT);
        }
    }
    unsigned landed = 0;
#define XREADY(q) ((unsigned)(XR0 + (4 * (q) + mt) * XRSPAN / 32 + XGATE))
    while ((landed = __builtin_amdgcn_readfirstlane(lds_ld(CTL))) < XREADY(0)) __builtin_amdgcn_s_sleep(8);
    const unsigned* fx = (const unsigned*)(myws + FX_OFF + (mt * 4 + wave) * 128) + (lane & 1);
    bool xok = wait_flags<16, FXSCOPE>(fx, call, tmo);
    if (wave == 0) STAMP(9);
    const unsigned char* abase = myws + slot + XST_OFF + (size_t)(mt * 8 + wave * 2) * 65536 + lane * 16;
#define AOFF(t, r, h) ((t) * 65536 + (r) * 2048 + (h) * 1024)
    constexpr int PF = 4;
    i32x4 a[PF][2][2];
#pragma unroll
    for (int u = 0; u < PF; ++u)
#pragma unroll
        for (int t = 0; t < 2; ++t)
#pragma unroll
            for (int h = 0; h < 2; ++h)
                a[u][t][h] = *(const i32x4*)(abase + AOFF(t, u, h));
    const float sc = p.sx[0] * p.sw[0];
    f32x4 acc[2][4];
#pragma unroll
    for (int t = 0; t < 2; ++t)
#pragma unroll
        for (int j = 0; j < 4; ++j) acc[t][j] = f32x4{0.f, 0.f, 0.f, 0.f};
    const unsigned char* bl = smem + (lane >> 4) * 2048 + (lane & 15) * 16;
#pragma unroll 1
    for (int r0 = 0; r0 < ROUNDS; r0 += PF) {
        const bool more = r0 + PF < ROUNDS;
        if (more) {
            while (landed < XREADY(r0 / PF + 1)) {
                landed = __builtin_amdgcn_readfirstlane(lds_ld(CTL));
                if (landed < XREADY(r0 / PF + 1)) __builtin_amdgcn_s_sleep(4);
            }
            xok = wait_flags<16, FXSCOPE>(fx + (r0 / PF + 1) * 2, call, tmo) && xok;
        }
#pragma unroll
        for (int u = 0; u < PF; ++u) {
            const int r = r0 + u;
            while (landed <= (unsigned)r) {
                landed = __builtin_amdgcn_readfirstlane(lds_ld(CTL));
                if (landed <= (unsigned)r) __builtin_amdgcn_s_sleep(1);
            }
            if (wave == 0) STAMP(16 + r);
            const unsigned char* bs = bl + (r % RING) * 8192;
            i32x8 bf[4];
#pragma unroll
            for (int j = 0; j < 4; ++j) {
                const i32x4 lo = *(const i32x4*)(bs + j * 256);
                const i32x4 hi = *(const i32x4*)(bs + 1024 + j * 256);
                bf[j] = __builtin_shufflevector(lo, hi, 0, 1, 2, 3, 4, 5, 6, 7);
            }
            lds_st(CTL + 4 + wave * 4, (unsigned)r + 1u);
#pragma unroll
            for (int t = 0; t < 2; ++t) {
                const i32x8 af = __builtin_shufflevector(a[u][t][0], a[u][t][1], 0, 1, 2, 3, 4, 5, 6, 7);
#pragma unroll
                for (int j = 0; j < 4; ++j)
                    acc[t][j] = __builtin_amdgcn_mfma_scale_f32_16x16x128_f8f6f4(af, bf[j], acc[t][j], 0, 0,
                                                                                 0, 0x7F7F7F7F, 0, 0x7F7F7F7F);
            }
            const int rn = more ? r + PF : r;
#pragma unroll
            for (int t = 0; t < 2; ++t)
#pragma unroll
                for (int h = 0; h < 2; ++h)
                    a[u][t][h] = *(const i32x4*)(abase + AOFF(t, rn, h));
        }
    }
    if (wave == 0) STAMP(10);
    const bool dead = !xok || landed != (unsigned)ROUNDS;
    const float scy = dead ? __builtin_nanf("") : sc;
    const float sce = -1.44269504088896341f * sc;
    constexpr float LIM = 60.f * 1.44269504088896341f;
    float* orow = p.out + (size_t)(mt * 128 + wave * 32 + (lane >> 4) * 4) * NN + g * 64 + (lane & 15) * 4;
#pragma unroll
    for (int t = 0; t < 2; ++t)
#pragma unroll
        for (int i = 0; i < 4; ++i) {
            f32x4 o;
#pragma unroll
            for (int j = 0; j < 4; ++j) {
                const float e = __builtin_amdgcn_exp2f(__builtin_amdgcn_fmed3f(acc[t][j][i] * sce, -LIM, LIM));
                o[j] = (acc[t][j][i] * scy) * __builtin_amdgcn_rcpf(1.f + e);
            }
            asm volatile("global_store_dwordx4 %0, %1, off sc1\n\ts_nop 1" :: "v"(orow + (size_t)(t * 16 + i) * NN), "v"(o) : "memory");
        }
}

}

extern "C" void kernel_launch(int rank, int nranks, ncclComm_t comm, hipStream_t stream,
                              void* const* d_in, const int* in_sizes, int n_in,
                              void* d_out, int out_size, const int* in_dtypes, int out_dtype,
                              void* const* d_out_all, void* const* d_ws_all, size_t ws_bytes) {
    (void)comm; (void)in_sizes; (void)n_in; (void)out_size; (void)in_dtypes; (void)out_dtype; (void)d_out_all;
    if (nranks != NR || ws_bytes < WS_NEED || (unsigned)rank >= (unsigned)NR) {
        fprintf(stderr, "unexpected configuration\n");
        return;
    }
    Params p;
    p.x = (const unsigned char*)d_in[0];
    p.w = (const unsigned char*)d_in[1];
    p.sx = (const float*)d_in[2];
    p.sw = (const float*)d_in[3];
    p.out = (float*)d_out;
    for (int i = 0; i < NR; ++i) p.ws[i] = (unsigned char*)d_ws_all[i];
    p.myws = p.ws[rank];
    p.rank = rank;
    gemm_gather_silu<<<NWG, 384, 0, stream>>>(p);
    HIP_CHECK(hipGetLastError());
}
```

```cpp
#include <hip/hip_runtime.h>
#include <rccl/rccl.h>
#include <cstdio>

#define HIP_CHECK(expr) do { hipError_t _e = (expr); if (_e != hipSuccess) { \
    fprintf(stderr, "HIP error %s at %s:%d\n", hipGetErrorString(_e), __FILE__, __LINE__); \
    return; } } while (0)

namespace {

constexpr int NR = 8;
constexpr int MS = 512;
constexpr int KS = 512;
constexpr int NN = 4096;
constexpr int NWG = 256;
constexpr int ROUNDS = KS / 16;
constexpr int RING = 16;
#define PACE 107
#define CATCH 4
#define LAG 90
#define TF 380
#define PT0 (TF + 40)
#define SDELAY 0
#define XW 1
#define XTRIG 2
#define XPAT 0
#define XMIX 0
#define SOFTR 0
#define SOFTT 200
#define MW 8
constexpr int RT = 8 / MW;
#define XSH 0
constexpr int WIN = 8;

constexpr int GS = 32768;
constexpr int XSS = 8192 + 256;
constexpr size_t WST_BYTES = (size_t)64 * GS;
constexpr size_t XST_BYTES = (size_t)32 * NR * XSS;
constexpr size_t XST_OFF = WST_BYTES;
constexpr size_t SLOT_BYTES = WST_BYTES + XST_BYTES;
constexpr size_t FW_OFF = 2 * SLOT_BYTES;
constexpr size_t AC_OFF = FW_OFF + 32 * 8 * 128;
constexpr size_t FX_OFF = AC_OFF + 32 * 128;
constexpr size_t CNT_OFF = FX_OFF + 16 * 128;
constexpr size_t TMO_OFF = CNT_OFF + NWG * 128;
constexpr size_t INV_OFF = TMO_OFF + 128;
constexpr size_t WS_NEED = INV_OFF + 16 * 256;
constexpr unsigned long long DEADLINE_TICKS = 50000000ull;

#define GAS __attribute__((address_space(1)))
typedef GAS unsigned char gchar;
typedef GAS unsigned gu32;
typedef GAS unsigned long long gu64;
typedef unsigned int u32x4 __attribute__((ext_vector_type(4)));
typedef int i32x8 __attribute__((ext_vector_type(8)));
typedef int i32x4 __attribute__((ext_vector_type(4)));
typedef float f32x4 __attribute__((ext_vector_type(4)));

#define STAMP(i) do {} while (0)

struct Params {
    const unsigned char* x;
    const unsigned char* w;
    unsigned char* myws;
    int rank;
    int pad;
    const float* sx;
    const float* sw;
    float* out;
    unsigned char* pad2;
    unsigned char* ws[NR];
};

__device__ __forceinline__ void store16_sys(gchar* p, u32x4 v) {
    asm volatile("global_store_dwordx4 %0, %1, off sc0 sc1\n\ts_nop 1" :: "v"(p), "v"(v) : "memory");
}
__device__ __forceinline__ void lds_st(unsigned addr, unsigned v) {
    asm volatile("ds_write_b32 %0, %1" :: "v"(addr), "v"(v) : "memory");
}
__device__ __forceinline__ unsigned lds_ld(unsigned addr) {
    unsigned v;
    asm volatile("ds_read_b32 %0, %1\n\ts_waitcnt lgkmcnt(0)" : "=v"(v) : "v"(addr) : "memory");
    return v;
}
__device__ __forceinline__ unsigned lds_ticket(unsigned addr, int lane) {
    unsigned old;
    asm volatile("ds_add_rtn_u32 %0, %1, %2\n\ts_waitcnt lgkmcnt(0)" : "=v"(old) : "v"(addr), "v"(lane == 0 ? 1u : 0u) : "memory");
    return __builtin_amdgcn_readfirstlane(old);
}
__device__ __forceinline__ void wait_until(unsigned t0, unsigned ticks) {
    while ((unsigned)__builtin_amdgcn_s_memrealtime() - t0 < ticks) __builtin_amdgcn_s_sleep(1);
}
__device__ __forceinline__ void glds16(const gchar* sbase, unsigned voff, unsigned lds_dst) {
    unsigned keep;
    asm volatile("s_mov_b32 %0, m0\n\ts_mov_b32 m0, %3\n\ts_nop 0\n\tglobal_load_lds_dwordx4 %1, %2\n\ts_mov_b32 m0, %0"
                 : "=&s"(keep) : "v"(voff), "s"(sbase), "s"(lds_dst) : "memory");
}
template <int N> __device__ __forceinline__ void wait_vm() { asm volatile("s_waitcnt vmcnt(%0)" :: "i"(N) : "memory"); }

__device__ __forceinline__ bool give_up(gu32* tmo, unsigned call, unsigned long long t0) {
    if (__hip_atomic_load(tmo, __ATOMIC_RELAXED, __HIP_MEMORY_SCOPE_AGENT) == call
        || __builtin_amdgcn_s_memrealtime() - t0 > DEADLINE_TICKS) {
        __hip_atomic_store(tmo, call, __ATOMIC_RELAXED, __HIP_MEMORY_SCOPE_AGENT);
        return true;
    }
    return false;
}
template <int SLEEP>
__device__ __forceinline__ bool wait_flags(const gu32* flag, unsigned want, unsigned call, gu32* tmo) {
    const unsigned long long t0 = __builtin_amdgcn_s_memrealtime();
    for (unsigned spin = 1;; ++spin) {
        const unsigned v = __hip_atomic_load(flag, __ATOMIC_RELAXED, __HIP_MEMORY_SCOPE_SYSTEM);
        if (__all(v >= want)) break;
        if ((spin & 63u) == 0u && give_up(tmo, call, t0)) return false;
        __builtin_amdgcn_s_sleep(SLEEP);
    }
    asm volatile("" ::: "memory");
    return true;
}

__global__ void __launch_bounds__((MW + 1) * 64, 1) gemm_gather_silu(Params p) {
    __shared__ __attribute__((aligned(16))) unsigned char smem[RING * 8192 + 64];
    const int tid = threadIdx.x, lane = tid & 63;
    const int wave = __builtin_amdgcn_readfirstlane(tid >> 6);
    const int b = blockIdx.x;
    const int g = (b & 7) * 8 + (b >> 5);
    const int mt = (b >> 3) & 3;
    const int xq = (b + XSH) & 7;
    const int gp = b >> 3;
    const unsigned lds0 = (unsigned)(size_t)(__attribute__((address_space(3))) unsigned char*)smem;
    const unsigned CTL = lds0 + RING * 8192;

    const unsigned t_start = (unsigned)__builtin_amdgcn_s_memrealtime();
    int rank = p.rank;
    gchar* myws = (gchar*)p.myws;
    const gchar* xin = (const gchar*)p.x;
    const gchar* win = (const gchar*)p.w;
    const int xd = (XMIX && xq) ? 1 + (xq - 1 + gp) % 7 : xq;
    gchar* dws = (gchar*)p.ws[xd];
    asm volatile("" : "+s"(xin), "+s"(win), "+s"(dws), "+s"(rank), "+s"(myws));
    if (wave == 0) STAMP(0);
    gu32* const tmo = (gu32*)(myws + TMO_OFF);
    gu32* const cnt = (gu32*)(myws + CNT_OFF + (size_t)b * 128) + wave;
    const unsigned cnt_old = __hip_atomic_load(cnt, __ATOMIC_RELAXED, __HIP_MEMORY_SCOPE_AGENT);
    if (tid < 16) lds_st(CTL + tid * 4, 0u);
#define INIT_BARRIER() asm volatile("s_waitcnt lgkmcnt(0)\n\ts_barrier" ::: "memory")
    unsigned call;
    size_t slot;
#define GET_CALL() do { asm volatile("v_readfirstlane_b32 %0, %1\n\ts_nop 3" : "=s"(call) : "v"(cnt_old)); call += 1u; \
        slot = (size_t)(call & 1u) * SLOT_BYTES; } while (0)
#define PUT_CALL() do { if (lane == 0) __hip_atomic_store(cnt, call, __ATOMIC_RELAXED, __HIP_MEMORY_SCOPE_AGENT); } while (0)
    bool ok = true;
    constexpr unsigned NARR = XTRIG ? 4 : 2;
    gu32* const ac = (gu32*)(myws + AC_OFF + gp * 128);

    if (wave < 2) {
        if (SDELAY) __builtin_amdgcn_s_sleep(SDELAY);
        const int cg = lane & 15, sg = gp * 2 + ((lane >> 4) & 1);
        const int r = (b & 7) * 4 + wave * 2 + (lane >> 5);
        const gchar* src = win + (size_t)(r * 16) * NN + sg * 64 + cg * 4;
        unsigned d[16];
#pragma unroll
        for (int i = 0; i < 16; ++i) d[i] = *(const gu32*)(src + (size_t)i * NN);
        INIT_BARRIER();
        GET_CALL();
        u32x4 o[4];
#pragma unroll
        for (int q = 0; q < 4; ++q) {
            const unsigned t0 = __builtin_amdgcn_perm(d[4 * q + 1], d[4 * q], 0x05010400u);
            const unsigned t1 = __builtin_amdgcn_perm(d[4 * q + 1], d[4 * q], 0x07030602u);
            const unsigned t2 = __builtin_amdgcn_perm(d[4 * q + 3], d[4 * q + 2], 0x05010400u);
            const unsigned t3 = __builtin_amdgcn_perm(d[4 * q + 3], d[4 * q + 2], 0x07030602u);
            o[0][q] = __builtin_amdgcn_perm(t2, t0, 0x05040100u);
            o[1][q] = __builtin_amdgcn_perm(t2, t0, 0x07060302u);
            o[2][q] = __builtin_amdgcn_perm(t3, t1, 0x05040100u);
            o[3][q] = __builtin_amdgcn_perm(t3, t1, 0x07060302u);
        }
        gchar* dst = myws + slot + (size_t)sg * GS + r * 1024 + cg * 16;
#pragma unroll
        for (int j = 0; j < 4; ++j) store16_sys(dst + j * 256, o[j]);
        if (wave == 0) STAMP(1);
        asm volatile("s_waitcnt vmcnt(0)" ::: "memory");
        if (wave == 0) STAMP(2);
        if (lds_ticket(CTL + 4, lane) == NARR - 1 && lane == 0)
            __hip_atomic_fetch_add(ac, 1u, __ATOMIC_RELAXED, __HIP_MEMORY_SCOPE_AGENT);
        if (xq == 0 && wave == 0) {
            gchar* peer_l = (gchar*)p.ws[lane & 7];
            wait_until(t_start, TF - 60);
            ok = wait_flags<2>(ac, call * 8u, call, tmo);
            STAMP(13);
            wait_until(t_start, TF);
            if (ok && lane < NR)
                __hip_atomic_store((gu32*)(peer_l + FW_OFF + (gp * 8 + rank) * 128), call,
                                   __ATOMIC_RELAXED, __HIP_MEMORY_SCOPE_SYSTEM);
        }
        PUT_CALL();
    } else if (wave < 4) {
        const unsigned dest = (unsigned)(rank + xd) & 7u;
        const gchar* src = xin + (size_t)(dest * MS + gp * 16 + (lane & 15)) * KS + (wave - 2) * 256 + (lane >> 4) * 16;
        u32x4 v[4];
#pragma unroll
        for (int i = 0; i < 4; ++i) v[i] = *(const GAS u32x4*)(src + i * 64);
        INIT_BARRIER();
        GET_CALL();
        gchar* dst = dws + slot + XST_OFF + (size_t)(gp * NR + rank) * XSS + (wave - 2) * 4096 + lane * 16;
#pragma unroll
        for (int i = 0; i < 4; ++i) asm volatile("" : "+v"(v[i]));
#pragma unroll
        for (int i = 0; i < 4; ++i) {
            store16_sys(dst + i * 1024, v[i]);
            if (i == 0 && wave == 2) STAMP(4);
            if (XW < 4 && !((XPAT >> i) & 1)) wait_vm<XW - 1>();
            if (XTRIG && i + 2 - XW == XTRIG) {
                if (lds_ticket(CTL + 4, lane) == NARR - 1 && lane == 0)
                    asm volatile("global_atomic_add %0, %1, off sc1" :: "v"(ac), "v"(1u) : "memory");
            }
        }
        asm volatile("s_waitcnt vmcnt(0)" ::: "memory");
        if (wave == 2) STAMP(5);
        if (lds_ticket(CTL + 8, lane) == 1u && lane == 0)
            __hip_atomic_store((gu32*)(dws + FX_OFF + (gp >> 1) * 128) + (gp & 1) * 8 + rank, call,
                               __ATOMIC_RELAXED, __HIP_MEMORY_SCOPE_SYSTEM);
        PUT_CALL();
    } else if (wave > 4) {
        INIT_BARRIER();
        GET_CALL();
        PUT_CALL();
    } else {
        __builtin_amdgcn_s_setprio(3);
        const unsigned xcc = (unsigned)__builtin_amdgcn_s_getreg((3 << 11) | 20) & 0xFu;
        gu32* const iv = (gu32*)(myws + INV_OFF + xcc * 256);
        bool inv_done = false;
        if (b < 8) asm volatile("buffer_inv sc0 sc1" ::: "memory");
        INIT_BARRIER();
        if (b < 8) {
            asm volatile("s_waitcnt vmcnt(0)" ::: "memory");
            inv_done = true;
        }
        GET_CALL();
        if (b < 8 && lane == 0) __hip_atomic_store(iv, call, __ATOMIC_RELAXED, __HIP_MEMORY_SCOPE_AGENT);
        PUT_CALL();
        const gchar* base[NR];
        unsigned ldst[NR];
#pragma unroll
        for (int i = 0; i < NR; ++i) {
            const unsigned rel = (unsigned)(g + i) & 7u;
            base[i] = (const gchar*)p.ws[rel] + slot;
            ldst[i] = lds0 + ((rel + rank) & 7u) * 1024;
        }
        STAMP(6);
        {
            const gu32* fw = lane < NR ? (const gu32*)(myws + FW_OFF + ((g >> 1) * 8 + lane) * 128) : iv;
            bool seen = lane > NR || (lane == NR && inv_done);
            wait_until(t_start, PT0);
            const unsigned long long t0 = __builtin_amdgcn_s_memrealtime();
            for (unsigned spin = 1;; ++spin) {
                if (!seen) seen = __hip_atomic_load(fw, __ATOMIC_RELAXED, __HIP_MEMORY_SCOPE_SYSTEM) >= call;
                const unsigned long long rdy = __ballot(seen);
                inv_done = inv_done || ((rdy >> NR) & 1);
                if ((rdy & 0xFFull) == 0xFFull && inv_done) break;
                if ((spin & 31u) == 0u) {
                    if (!inv_done) {
                        asm volatile("buffer_inv sc0 sc1\n\ts_waitcnt vmcnt(0)" ::: "memory");
                        inv_done = true;
                    }
                    if (give_up(tmo, call, t0)) { ok = false; break; }
                }
                __builtin_amdgcn_s_sleep(1);
            }
        }
        const unsigned long long t7 = __builtin_amdgcn_s_memrealtime();
        const unsigned tref = (unsigned)t7;
        unsigned voff = g * GS + lane * 16;
#pragma unroll 1
        for (int r = 0; r < ROUNDS && ok; ++r) {
            const unsigned sched = (unsigned)(r < SOFTR ? SOFTT * r : SOFTT * SOFTR + PACE * (r - SOFTR));
            const unsigned now = (unsigned)__builtin_amdgcn_s_memrealtime() - tref;
            const bool behind = now > sched + LAG;
            if (now < sched) wait_until(tref, sched);
            if (r >= RING) {
                for (;;) {
                    const unsigned dn = lds_ld(CTL + 32 + (lane & (MW - 1)) * 4);
                    if (__all(dn + RING > (unsigned)r)) break;
                    wait_vm<0>();
                    lds_st(CTL, (unsigned)r);
                    __builtin_amdgcn_s_sleep(1);
                }
            }
            const unsigned so = (unsigned)(r % RING) * 8192;
            if (CATCH && behind) {
#pragma unroll
                for (int i = 0; i < NR; ++i) {
                    glds16(base[i], voff, ldst[i] + so);
                    wait_vm<WIN - 1 + CATCH>();
                }
                wait_vm<8>();
                lds_st(CTL, (unsigned)r);
            } else {
#pragma unroll
                for (int i = 0; i < NR; ++i) {
                    if (SOFTR && r < SOFTR && i) wait_until(tref, sched + i * (SOFTT / 8));
                    glds16(base[i], voff, ldst[i] + so);
                    wait_vm<WIN - 1>();
                    if (i == NR - 2) lds_st(CTL, (unsigned)r);
                }
            }
            voff += 1024;
        }
        wait_vm<0>();
        lds_st(CTL, ok ? (unsigned)ROUNDS : 0x80000000u);
        STAMP(8);
        return;
    }

    unsigned landed = 0;
    while ((landed = __builtin_amdgcn_readfirstlane(lds_ld(CTL))) < 2u) __builtin_amdgcn_s_sleep(8);
    const int mw = wave > 4 ? wave - 1 : wave;
    const int rt0 = mt * 8 + mw * RT;
    ok = wait_flags<16>((const gu32*)(myws + FX_OFF + (rt0 >> 1) * 128) + (rt0 & 1) * 8 + (lane & (RT * 8 - 1)), call, call, tmo) && ok;
    if (wave == 0) STAMP(9);
    const gchar* abase = myws + slot + XST_OFF + (size_t)(rt0 * NR + (lane >> 4) * 2) * XSS + (lane & 15) * 16;
    constexpr int PF = 4;
    i32x4 a[PF][RT][2];
#pragma unroll
    for (int u = 0; u < PF; ++u)
#pragma unroll
        for (int t = 0; t < RT; ++t)
#pragma unroll
            for (int h = 0; h < 2; ++h)
                a[u][t][h] = *(const GAS i32x4*)(abase + (t * NR + h) * XSS + u * 256);
    float sc = p.sx[0] * p.sw[0];
    float sce = -1.44269504088896341f * sc;
    GAS float* orow = (GAS float*)p.out + (size_t)(rt0 * 16 + (lane >> 4) * 4) * NN + g * 64 + (lane & 15) * 4;
    asm volatile("" : "+v"(sc), "+v"(sce), "+v"(orow));
    f32x4 acc[RT][4];
#pragma unroll
    for (int t = 0; t < RT; ++t)
#pragma unroll
        for (int j = 0; j < 4; ++j) acc[t][j] = f32x4{0.f, 0.f, 0.f, 0.f};
    const unsigned char* bl = smem + (lane >> 4) * 2048 + (lane & 15) * 16;
#pragma unroll 1
    for (int r0 = 0; r0 < ROUNDS; r0 += PF) {
        const bool more = r0 + PF < ROUNDS;
#pragma unroll
        for (int u = 0; u < PF; ++u) {
            const int r = r0 + u;
            while (landed <= (unsigned)r) {
                landed = __builtin_amdgcn_readfirstlane(lds_ld(CTL));
                if (landed <= (unsigned)r) __builtin_amdgcn_s_sleep(1);
            }
            if (wave == 0) STAMP(16 + r);
            const unsigned char* bs = bl + (r % RING) * 8192;
            i32x8 bf[4];
#pragma unroll
            for (int j = 0; j < 4; ++j) {
                const i32x4 lo = *(const i32x4*)(bs + j * 256);
                const i32x4 hi = *(const i32x4*)(bs + 1024 + j * 256);
                bf[j] = __builtin_shufflevector(lo, hi, 0, 1, 2, 3, 4, 5, 6, 7);
            }
            lds_st(CTL + 32 + mw * 4, (unsigned)r + 1u);
            __builtin_amdgcn_sched_barrier(0);
#pragma unroll
            for (int t = 0; t < RT; ++t) {
                const i32x8 af = __builtin_shufflevector(a[u][t][0], a[u][t][1], 0, 1, 2, 3, 4, 5, 6, 7);
#pragma unroll
                for (int j = 0; j < 4; ++j)
                    acc[t][j] = __builtin_amdgcn_mfma_scale_f32_16x16x128_f8f6f4(af, bf[j], acc[t][j], 0, 0,
                                                                                 0, 0x7F7F7F7F, 0, 0x7F7F7F7F);
                asm volatile("" : "+v"(acc[t][0]), "+v"(acc[t][1]), "+v"(acc[t][2]), "+v"(acc[t][3]));
            }
            const int rn = more ? r + PF : r;
#pragma unroll
            for (int t = 0; t < RT; ++t)
#pragma unroll
                for (int h = 0; h < 2; ++h)
                    a[u][t][h] = *(const GAS i32x4*)(abase + (t * NR + h) * XSS + rn * 256);
        }
    }
    if (wave == 0) STAMP(10);
    const bool dead = !ok || landed != (unsigned)ROUNDS;
    const float scy = dead ? __builtin_nanf("") : sc;
    constexpr float LIM = 60.f * 1.44269504088896341f;
#pragma unroll
    for (int t = 0; t < RT; ++t)
#pragma unroll
        for (int i = 0; i < 4; ++i) {
            f32x4 o;
#pragma unroll
            for (int j = 0; j < 4; ++j) {
                const float e = __builtin_amdgcn_exp2f(__builtin_amdgcn_fmed3f(acc[t][j][i] * sce, -LIM, LIM));
                o[j] = (acc[t][j][i] * scy) * __builtin_amdgcn_rcpf(1.f + e);
            }
            asm volatile("global_store_dwordx4 %0, %1, off sc1\n\ts_nop 1" :: "v"(orow + (size_t)(t * 16 + i) * NN), "v"(o) : "memory");
        }
}

}

extern "C" void kernel_launch(int rank, int nranks, ncclComm_t comm, hipStream_t stream,
                              void* const* d_in, const int* in_sizes, int n_in,
                              void* d_out, int out_size, const int* in_dtypes, int out_dtype,
                              void* const* d_out_all, void* const* d_ws_all, size_t ws_bytes) {
    (void)comm; (void)in_sizes; (void)n_in; (void)out_size; (void)in_dtypes; (void)out_dtype; (void)d_out_all;
    if (nranks != NR || ws_bytes < WS_NEED || (unsigned)rank >= (unsigned)NR) {
        fprintf(stderr, "unexpected configuration\n");
        return;
    }
    Params p;
    p.x = (const unsigned char*)d_in[0];
    p.w = (const unsigned char*)d_in[1];
    p.sx = (const float*)d_in[2];
    p.sw = (const float*)d_in[3];
    p.out = (float*)d_out;
    for (int i = 0; i < NR; ++i) p.ws[i] = (unsigned char*)d_ws_all[(rank + i) % NR];
    p.myws = p.ws[0];
    p.rank = rank;
    p.pad = 0;
    p.pad2 = nullptr;
    gemm_gather_silu<<<NWG, (MW + 1) * 64, 0, stream>>>(p);
    HIP_CHECK(hipGetLastError());
}
```

```cpp
#include <hip/hip_runtime.h>
#include <rccl/rccl.h>
#include <cstdio>

#define HIP_CHECK(expr) do { hipError_t _e = (expr); if (_e != hipSuccess) { \
    fprintf(stderr, "HIP error %s at %s:%d\n", hipGetErrorString(_e), __FILE__, __LINE__); \
    return; } } while (0)

namespace {

constexpr int NR = 8;
constexpr int MS = 512;
constexpr int KS = 512;
constexpr int NN = 4096;
constexpr int NWG = 256;
constexpr int MW = 8;
constexpr int ROUNDS = KS / 16;
constexpr int RING = 16;
constexpr int WIN = 8;
constexpr unsigned PACE = 107;
constexpr unsigned LAG = 90;
constexpr int CATCH = 4;
constexpr unsigned TF = 380;
constexpr unsigned PT0 = TF + 40;

constexpr int GS = 32768;
constexpr int XSS = 8192 + 256;
constexpr size_t WST_BYTES = (size_t)64 * GS;
constexpr size_t XST_BYTES = (size_t)32 * NR * XSS;
constexpr size_t XST_OFF = WST_BYTES;
constexpr size_t SLOT_BYTES = WST_BYTES + XST_BYTES;
constexpr size_t FW_OFF = 2 * SLOT_BYTES;
constexpr size_t AC_OFF = FW_OFF + 32 * 8 * 128;
constexpr size_t FX_OFF = AC_OFF + 32 * 128;
constexpr size_t CNT_OFF = FX_OFF + 16 * 128;
constexpr size_t TMO_OFF = CNT_OFF + NWG * 128;
constexpr size_t INV_OFF = TMO_OFF + 128;
constexpr size_t WS_NEED = INV_OFF + 16 * 256;
constexpr unsigned long long DEADLINE_TICKS = 50000000ull;

#define GAS __attribute__((address_space(1)))
typedef GAS unsigned char gchar;
typedef GAS unsigned gu32;
typedef unsigned int u32x4 __attribute__((ext_vector_type(4)));
typedef int i32x8 __attribute__((ext_vector_type(8)));
typedef int i32x4 __attribute__((ext_vector_type(4)));
typedef float f32x4 __attribute__((ext_vector_type(4)));

struct Params {
    const unsigned char* x;
    const unsigned char* w;
    unsigned char* myws;
    int rank;
    int pad;
    const float* sx;
    const float* sw;
    float* out;
    unsigned char* pad2;
    unsigned char* ws[NR];
};

__device__ __forceinline__ void store16_sys(gchar* p, u32x4 v) {
    asm volatile("global_store_dwordx4 %0, %1, off sc0 sc1\n\ts_nop 1" :: "v"(p), "v"(v) : "memory");
}
__device__ __forceinline__ void lds_st(unsigned addr, unsigned v) {
    asm volatile("ds_write_b32 %0, %1" :: "v"(addr), "v"(v) : "memory");
}
__device__ __forceinline__ unsigned lds_ld(unsigned addr) {
    unsigned v;
    asm volatile("ds_read_b32 %0, %1\n\ts_waitcnt lgkmcnt(0)" : "=v"(v) : "v"(addr) : "memory");
    return v;
}
__device__ __forceinline__ unsigned lds_ticket(unsigned addr, int lane) {
    unsigned old;
    asm volatile("ds_add_rtn_u32 %0, %1, %2\n\ts_waitcnt lgkmcnt(0)"
                 : "=v"(old) : "v"(addr), "v"(lane == 0 ? 1u : 0u) : "memory");
    return __builtin_amdgcn_readfirstlane(old);
}
__device__ __forceinline__ void wait_until(unsigned t0, unsigned ticks) {
    while ((unsigned)__builtin_amdgcn_s_memrealtime() - t0 < ticks) __builtin_amdgcn_s_sleep(1);
}
__device__ __forceinline__ void glds16(const gchar* sbase, unsigned voff, unsigned lds_dst) {
    unsigned keep;
    asm volatile("s_mov_b32 %0, m0\n\ts_mov_b32 m0, %3\n\ts_nop 0\n\tglobal_load_lds_dwordx4 %1, %2\n\ts_mov_b32 m0, %0"
                 : "=&s"(keep) : "v"(voff), "s"(sbase), "s"(lds_dst) : "memory");
}
template <int N> __device__ __forceinline__ void wait_vm() { asm volatile("s_waitcnt vmcnt(%0)" :: "i"(N) : "memory"); }

__device__ __forceinline__ bool give_up(gu32* tmo, unsigned call, unsigned long long t0) {
    if (__hip_atomic_load(tmo, __ATOMIC_RELAXED, __HIP_MEMORY_SCOPE_AGENT) == call
        || __builtin_amdgcn_s_memrealtime() - t0 > DEADLINE_TICKS) {
        __hip_atomic_store(tmo, call, __ATOMIC_RELAXED, __HIP_MEMORY_SCOPE_AGENT);
        return true;
    }
    return false;
}
template <int SLEEP>
__device__ __forceinline__ bool wait_flags(const gu32* flag, unsigned want, unsigned call, gu32* tmo) {
    const unsigned long long t0 = __builtin_amdgcn_s_memrealtime();
    for (unsigned spin = 1;; ++spin) {
        const unsigned v = __hip_atomic_load(flag, __ATOMIC_RELAXED, __HIP_MEMORY_SCOPE_SYSTEM);
        if (__all(v >= want)) break;
        if ((spin & 63u) == 0u && give_up(tmo, call, t0)) return false;
        __builtin_amdgcn_s_sleep(SLEEP);
    }
    asm volatile("" ::: "memory");
    return true;
}

__global__ void __launch_bounds__((MW + 1) * 64, 1) gemm_gather_silu(Params p) {
    __shared__ __attribute__((aligned(16))) unsigned char smem[RING * 8192 + 64];
    const int tid = threadIdx.x, lane = tid & 63;
    const int wave = __builtin_amdgcn_readfirstlane(tid >> 6);
    const int b = blockIdx.x;
    const int g = (b & 7) * 8 + (b >> 5);
    const int mt = (b >> 3) & 3;
    const int xq = b & 7;
    const int gp = b >> 3;
    const int xd = xq ? 1 + (xq - 1 + gp) % 7 : 0;
    const unsigned lds0 = (unsigned)(size_t)(__attribute__((address_space(3))) unsigned char*)smem;
    const unsigned CTL = lds0 + RING * 8192;

    const unsigned t_start = (unsigned)__builtin_amdgcn_s_memrealtime();
    int rank = p.rank;
    gchar* myws = (gchar*)p.myws;
    const gchar* xin = (const gchar*)p.x;
    const gchar* win = (const gchar*)p.w;
    gchar* dws = (gchar*)p.ws[xd];
    asm volatile("" : "+s"(xin), "+s"(win), "+s"(dws), "+s"(rank), "+s"(myws));
    gu32* const tmo = (gu32*)(myws + TMO_OFF);
    gu32* const cnt = (gu32*)(myws + CNT_OFF + (size_t)b * 128) + wave;
    const unsigned cnt_old = __hip_atomic_load(cnt, __ATOMIC_RELAXED, __HIP_MEMORY_SCOPE_AGENT);
    if (tid < 16) lds_st(CTL + tid * 4, 0u);
#define INIT_BARRIER() asm volatile("s_waitcnt lgkmcnt(0)\n\ts_barrier" ::: "memory")
    unsigned call;
    size_t slot;
#define GET_CALL() do { asm volatile("v_readfirstlane_b32 %0, %1\n\ts_nop 3" : "=s"(call) : "v"(cnt_old)); call += 1u; \
        slot = (size_t)(call & 1u) * SLOT_BYTES; } while (0)
#define PUT_CALL() do { if (lane == 0) __hip_atomic_store(cnt, call, __ATOMIC_RELAXED, __HIP_MEMORY_SCOPE_AGENT); } while (0)
    bool ok = true;

    if (wave < 2) {
        const unsigned dest = (unsigned)(rank + xd) & 7u;
        const gchar* src = xin + (size_t)(dest * MS + gp * 16 + (lane & 15)) * KS + wave * 256 + (lane >> 4) * 16;
        u32x4 v[4];
#pragma unroll
        for (int i = 0; i < 4; ++i) v[i] = *(const GAS u32x4*)(src + i * 64);
        INIT_BARRIER();
        GET_CALL();
        gchar* dst = dws + slot + XST_OFF + (size_t)(gp * NR + rank) * XSS + wave * 4096 + lane * 16;
#pragma unroll
        for (int i = 0; i < 4; ++i) {
            store16_sys(dst + i * 1024, v[i]);
            wait_vm<0>();
        }
        if (lds_ticket(CTL + 8, lane) == 1u && lane == 0)
            __hip_atomic_store((gu32*)(dws + FX_OFF + (gp >> 1) * 128) + (gp & 1) * 8 + rank, call,
                               __ATOMIC_RELAXED, __HIP_MEMORY_SCOPE_SYSTEM);
        PUT_CALL();
    } else if (wave < 4) {
        const int cg = lane & 15, sg = gp * 2 + ((lane >> 4) & 1);
        const int r = xq * 4 + (wave - 2) * 2 + (lane >> 5);
        const gchar* src = win + (size_t)(r * 16) * NN + sg * 64 + cg * 4;
        unsigned d[16];
#pragma unroll
        for (int i = 0; i < 16; ++i) d[i] = *(const gu32*)(src + (size_t)i * NN);
        INIT_BARRIER();
        GET_CALL();
        u32x4 o[4];
#pragma unroll
        for (int q = 0; q < 4; ++q) {
            const unsigned t0 = __builtin_amdgcn_perm(d[4 * q + 1], d[4 * q], 0x05010400u);
            const unsigned t1 = __builtin_amdgcn_perm(d[4 * q + 1], d[4 * q], 0x07030602u);
            const unsigned t2 = __builtin_amdgcn_perm(d[4 * q + 3], d[4 * q + 2], 0x05010400u);
            const unsigned t3 = __builtin_amdgcn_perm(d[4 * q + 3], d[4 * q + 2], 0x07030602u);
            o[0][q] = __builtin_amdgcn_perm(t2, t0, 0x05040100u);
            o[1][q] = __builtin_amdgcn_perm(t2, t0, 0x07060302u);
            o[2][q] = __builtin_amdgcn_perm(t3, t1, 0x05040100u);
            o[3][q] = __builtin_amdgcn_perm(t3, t1, 0x07060302u);
        }
        gchar* dst = myws + slot + (size_t)sg * GS + r * 1024 + cg * 16;
#pragma unroll
        for (int j = 0; j < 4; ++j) store16_sys(dst + j * 256, o[j]);
        wait_vm<0>();
        gu32* const ac = (gu32*)(myws + AC_OFF + gp * 128);
        if (lds_ticket(CTL + 4, lane) == 1u && lane == 0)
            __hip_atomic_fetch_add(ac, 1u, __ATOMIC_RELAXED, __HIP_MEMORY_SCOPE_AGENT);
        if (xq == 0 && wave == 2) {
            gchar* peer_l = (gchar*)p.ws[lane & 7];
            wait_until(t_start, TF - 60);
            ok = wait_flags<2>(ac, call * 8u, call, tmo);
            wait_until(t_start, TF);
            if (ok && lane < NR)
                __hip_atomic_store((gu32*)(peer_l + FW_OFF + (gp * 8 + rank) * 128), call,
                                   __ATOMIC_RELAXED, __HIP_MEMORY_SCOPE_SYSTEM);
        }
        PUT_CALL();
    } else if (wave > 4) {
        INIT_BARRIER();
        GET_CALL();
        PUT_CALL();
    } else {
        __builtin_amdgcn_s_setprio(3);
        const unsigned xcc = (unsigned)__builtin_amdgcn_s_getreg((3 << 11) | 20) & 0xFu;
        gu32* const iv = (gu32*)(myws + INV_OFF + xcc * 256);
        bool inv_done = false;
        if (b < 8) asm volatile("buffer_inv sc0 sc1" ::: "memory");
        INIT_BARRIER();
        if (b < 8) {
            wait_vm<0>();
            inv_done = true;
        }
        GET_CALL();
        if (b < 8 && lane == 0) __hip_atomic_store(iv, call, __ATOMIC_RELAXED, __HIP_MEMORY_SCOPE_AGENT);
        PUT_CALL();
        const gchar* base[NR];
        unsigned ldst[NR];
#pragma unroll
        for (int i = 0; i < NR; ++i) {
            const unsigned rel = (unsigned)(g + i) & 7u;
            base[i] = (const gchar*)p.ws[rel] + slot;
            ldst[i] = lds0 + ((rel + rank) & 7u) * 1024;
        }
        {
            const gu32* fw = lane < NR ? (const gu32*)(myws + FW_OFF + ((g >> 1) * 8 + lane) * 128) : iv;
            bool seen = lane > NR || (lane == NR && inv_done);
            wait_until(t_start, PT0);
            const unsigned long long t0 = __builtin_amdgcn_s_memrealtime();
            for (unsigned spin = 1;; ++spin) {
                if (!seen) seen = __hip_atomic_load(fw, __ATOMIC_RELAXED, __HIP_MEMORY_SCOPE_SYSTEM) >= call;
                const unsigned long long rdy = __ballot(seen);
                inv_done = inv_done || ((rdy >> NR) & 1);
                if ((rdy & 0xFFull) == 0xFFull && inv_done) break;
                if ((spin & 31u) == 0u) {
                    if (!inv_done) {
                        asm volatile("buffer_inv sc0 sc1\n\ts_waitcnt vmcnt(0)" ::: "memory");
                        inv_done = true;
                    }
                    if (give_up(tmo, call, t0)) { ok = false; break; }
                }
                __builtin_amdgcn_s_sleep(1);
            }
        }
        const unsigned tref = (unsigned)__builtin_amdgcn_s_memrealtime();
        unsigned voff = g * GS + lane * 16;
#pragma unroll 1
        for (int r = 0; r < ROUNDS && ok; ++r) {
            const unsigned sched = PACE * (unsigned)r;
            const unsigned now = (unsigned)__builtin_amdgcn_s_memrealtime() - tref;
            const bool behind = now > sched + LAG;
            if (now < sched) wait_until(tref, sched);
            if (r >= RING) {
                for (;;) {
                    const unsigned dn = lds_ld(CTL + 32 + (lane & (MW - 1)) * 4);
                    if (__all(dn + RING > (unsigned)r)) break;
                    wait_vm<0>();
                    lds_st(CTL, (unsigned)r);
                    __builtin_amdgcn_s_sleep(1);
                }
            }
            const unsigned so = (unsigned)(r % RING) * 8192;
            if (behind) {
#pragma unroll
                for (int i = 0; i < NR; ++i) {
                    glds16(base[i], voff, ldst[i] + so);
                    wait_vm<WIN - 1 + CATCH>();
                }
                wait_vm<NR>();
                lds_st(CTL, (unsigned)r);
            } else {
#pragma unroll
                for (int i = 0; i < NR; ++i) {
                    glds16(base[i], voff, ldst[i] + so);
                    wait_vm<WIN - 1>();
                    if (i == NR - 2) lds_st(CTL, (unsigned)r);
                }
            }
            voff += 1024;
        }
        wait_vm<0>();
        lds_st(CTL, ok ? (unsigned)ROUNDS : 0x80000000u);
        return;
    }

    unsigned landed = 0;
    while ((landed = __builtin_amdgcn_readfirstlane(lds_ld(CTL))) < 2u) __builtin_amdgcn_s_sleep(8);
    const int mw = wave > 4 ? wave - 1 : wave;
    const int rt = mt * 8 + mw;
    ok = wait_flags<16>((const gu32*)(myws + FX_OFF + (rt >> 1) * 128) + (rt & 1) * 8 + (lane & 7), call, call, tmo) && ok;
    const gchar* abase = myws + slot + XST_OFF + (size_t)(rt * NR + (lane >> 4) * 2) * XSS + (lane & 15) * 16;
    constexpr int PF = 4;
    i32x4 a[PF][2];
#pragma unroll
    for (int u = 0; u < PF; ++u)
#pragma unroll
        for (int h = 0; h < 2; ++h) a[u][h] = *(const GAS i32x4*)(abase + h * XSS + u * 256);
    float sc = p.sx[0] * p.sw[0];
    float sce = -1.44269504088896341f * sc;
    GAS float* orow = (GAS float*)p.out + (size_t)(rt * 16 + (lane >> 4) * 4) * NN + g * 64 + (lane & 15) * 4;
    asm volatile("" : "+v"(sc), "+v"(sce), "+v"(orow));
    f32x4 acc[4];
#pragma unroll
    for (int j = 0; j < 4; ++j) acc[j] = f32x4{0.f, 0.f, 0.f, 0.f};
    const unsigned char* bl = smem + (lane >> 4) * 2048 + (lane & 15) * 16;
#pragma unroll 1
    for (int r0 = 0; r0 < ROUNDS; r0 += PF) {
        const bool more = r0 + PF < ROUNDS;
#pragma unroll
        for (int u = 0; u < PF; ++u) {
            const int r = r0 + u;
            while (landed <= (unsigned)r) {
                landed = __builtin_amdgcn_readfirstlane(lds_ld(CTL));
                if (landed <= (unsigned)r) __builtin_amdgcn_s_sleep(1);
            }
            const unsigned char* bs = bl + (r % RING) * 8192;
            i32x8 bf[4];
#pragma unroll
            for (int j = 0; j < 4; ++j) {
                const i32x4 lo = *(const i32x4*)(bs + j * 256);
                const i32x4 hi = *(const i32x4*)(bs + 1024 + j * 256);
                bf[j] = __builtin_shufflevector(lo, hi, 0, 1, 2, 3, 4, 5, 6, 7);
            }
            lds_st(CTL + 32 + mw * 4, (unsigned)r + 1u);
            __builtin_amdgcn_sched_barrier(0);
            const i32x8 af = __builtin_shufflevector(a[u][0], a[u][1], 0, 1, 2, 3, 4, 5, 6, 7);
#pragma unroll
            for (int j = 0; j < 4; ++j)
                acc[j] = __builtin_amdgcn_mfma_scale_f32_16x16x128_f8f6f4(af, bf[j], acc[j], 0, 0,
                                                                          0, 0x7F7F7F7F, 0, 0x7F7F7F7F);
            asm volatile("" : "+v"(acc[0]), "+v"(acc[1]), "+v"(acc[2]), "+v"(acc[3]));
            const int rn = more ? r + PF : r;
#pragma unroll
            for (int h = 0; h < 2; ++h) a[u][h] = *(const GAS i32x4*)(abase + h * XSS + rn * 256);
        }
    }
    const bool dead = !ok || landed != (unsigned)ROUNDS;
    const float scy = dead ? __builtin_nanf("") : sc;
    constexpr float LIM = 60.f * 1.44269504088896341f;
#pragma unroll
    for (int i = 0; i < 4; ++i) {
        f32x4 o;
#pragma unroll
        for (int j = 0; j < 4; ++j) {
            const float e = __builtin_amdgcn_exp2f(__builtin_amdgcn_fmed3f(acc[j][i] * sce, -LIM, LIM));
            o[j] = (acc[j][i] * scy) * __builtin_amdgcn_rcpf(1.f + e);
        }
        asm volatile("global_store_dwordx4 %0, %1, off sc1\n\ts_nop 1" :: "v"(orow + (size_t)i * NN), "v"(o) : "memory");
    }
}

}

extern "C" void kernel_launch(int rank, int nranks, ncclComm_t comm, hipStream_t stream,
                              void* const* d_in, const int* in_sizes, int n_in,
                              void* d_out, int out_size, const int* in_dtypes, int out_dtype,
                              void* const* d_out_all, void* const* d_ws_all, size_t ws_bytes) {
    (void)comm; (void)in_dtypes; (void)out_dtype; (void)d_out_all;
    if (nranks != NR || (unsigned)rank >= (unsigned)NR || ws_bytes < WS_NEED || n_in != 4
        || in_sizes[0] != NR * MS * KS || in_sizes[1] != KS * NN || out_size != MS * NN) {
        fprintf(stderr, "unexpected configuration\n");
        return;
    }
    Params p;
    p.x = (const unsigned char*)d_in[0];
    p.w = (const unsigned char*)d_in[1];
    p.sx = (const float*)d_in[2];
    p.sw = (const float*)d_in[3];
    p.out = (float*)d_out;
    for (int i = 0; i < NR; ++i) p.ws[i] = (unsigned char*)d_ws_all[(rank + i) % NR];
    p.myws = p.ws[0];
    p.rank = rank;
    p.pad = 0;
    p.pad2 = nullptr;
    gemm_gather_silu<<<NWG, (MW + 1) * 64, 0, stream>>>(p);
    HIP_CHECK(hipGetLastError());
}
```

```cpp
#include <hip/hip_runtime.h>
#include <rccl/rccl.h>
#include <cstdio>

#define HIP_CHECK(expr) do { hipError_t _e = (expr); if (_e != hipSuccess) { \
    fprintf(stderr, "HIP error %s at %s:%d\n", hipGetErrorString(_e), __FILE__, __LINE__); \
    return; } } while (0)

namespace {

constexpr int NR = 8;
constexpr int MS = 512;
constexpr int KS = 512;
constexpr int NN = 4096;
constexpr int NWG = 256;
constexpr int MW = 8;
constexpr int ROUNDS = KS / 16;
constexpr int RING = 16;
constexpr int WIN = 8;
constexpr unsigned PACE = 107;
constexpr unsigned LAG = 90;
constexpr int CATCH = 4;
#define TFX 280
#define SDELAY 8
#define CODEPF 0
#define PT0 380
constexpr unsigned TF = TFX;

constexpr int GS = 32768;
constexpr int XSS = 8192 + 256;
constexpr size_t WST_BYTES = (size_t)64 * GS;
constexpr size_t XST_BYTES = (size_t)32 * NR * XSS;
constexpr size_t XST_OFF = WST_BYTES;
constexpr size_t SLOT_BYTES = WST_BYTES + XST_BYTES;
constexpr size_t FW_OFF = 2 * SLOT_BYTES;
constexpr size_t FA_OFF = FW_OFF + 32 * 8 * 128;
constexpr size_t AC_OFF = FA_OFF + 32 * 128;
constexpr size_t FX_OFF = AC_OFF + 32 * 128;
constexpr size_t CNT_OFF = FX_OFF + 16 * 128;
constexpr size_t TMO_OFF = CNT_OFF + NWG * 128;
constexpr size_t INV_OFF = TMO_OFF + 128;
constexpr size_t WS_NEED = INV_OFF + 16 * 256;
constexpr unsigned long long DEADLINE_TICKS = 50000000ull;

#define GAS __attribute__((address_space(1)))
typedef GAS unsigned char gchar;
typedef GAS unsigned gu32;
typedef unsigned int u32x4 __attribute__((ext_vector_type(4)));
typedef int i32x8 __attribute__((ext_vector_type(8)));
typedef int i32x4 __attribute__((ext_vector_type(4)));
typedef float f32x4 __attribute__((ext_vector_type(4)));

#define STAMP(i) do {} while (0)

struct Params {
    const unsigned char* x;
    const unsigned char* w;
    unsigned char* myws;
    int rank;
    int pad;
    const float* sx;
    const float* sw;
    float* out;
    unsigned char* pad2;
    unsigned char* ws[NR];
};

__device__ __forceinline__ void store16_sys(gchar* p, u32x4 v) {
    asm volatile("global_store_dwordx4 %0, %1, off sc0 sc1\n\ts_nop 1" :: "v"(p), "v"(v) : "memory");
}
__device__ __forceinline__ void lds_st(unsigned addr, unsigned v) {
    asm volatile("ds_write_b32 %0, %1" :: "v"(addr), "v"(v) : "memory");
}
__device__ __forceinline__ unsigned lds_ld(unsigned addr) {
    unsigned v;
    asm volatile("ds_read_b32 %0, %1\n\ts_waitcnt lgkmcnt(0)" : "=v"(v) : "v"(addr) : "memory");
    return v;
}
__device__ __forceinline__ unsigned lds_ticket(unsigned addr, int lane) {
    unsigned old;
    asm volatile("ds_add_rtn_u32 %0, %1, %2\n\ts_waitcnt lgkmcnt(0)"
                 : "=v"(old) : "v"(addr), "v"(lane == 0 ? 1u : 0u) : "memory");
    return __builtin_amdgcn_readfirstlane(old);
}
__device__ __forceinline__ void wait_until(unsigned t0, unsigned ticks) {
    while ((unsigned)__builtin_amdgcn_s_memrealtime() - t0 < ticks) __builtin_amdgcn_s_sleep(1);
}
__device__ __forceinline__ void glds16(const gchar* sbase, unsigned voff, unsigned lds_dst) {
    unsigned keep;
    asm volatile("s_mov_b32 %0, m0\n\ts_mov_b32 m0, %3\n\ts_nop 0\n\tglobal_load_lds_dwordx4 %1, %2\n\ts_mov_b32 m0, %0"
                 : "=&s"(keep) : "v"(voff), "s"(sbase), "s"(lds_dst) : "memory");
}
template <int N> __device__ __forceinline__ void wait_vm() { asm volatile("s_waitcnt vmcnt(%0)" :: "i"(N) : "memory"); }

__device__ __forceinline__ bool give_up(gu32* tmo, unsigned call, unsigned long long t0) {
    if (__hip_atomic_load(tmo, __ATOMIC_RELAXED, __HIP_MEMORY_SCOPE_AGENT) == call
        || __builtin_amdgcn_s_memrealtime() - t0 > DEADLINE_TICKS) {
        __hip_atomic_store(tmo, call, __ATOMIC_RELAXED, __HIP_MEMORY_SCOPE_AGENT);
        return true;
    }
    return false;
}
template <int SLEEP>
__device__ __forceinline__ bool wait_flags(const gu32* flag, unsigned want, unsigned call, gu32* tmo) {
    const unsigned long long t0 = __builtin_amdgcn_s_memrealtime();
    for (unsigned spin = 1;; ++spin) {
        const unsigned v = __hip_atomic_load(flag, __ATOMIC_RELAXED, __HIP_MEMORY_SCOPE_SYSTEM);
        if (__all(v >= want)) break;
        if ((spin & 63u) == 0u && give_up(tmo, call, t0)) return false;
        __builtin_amdgcn_s_sleep(SLEEP);
    }
    asm volatile("" ::: "memory");
    return true;
}

__global__ void __launch_bounds__((MW + 1) * 64, 1) gemm_gather_silu(Params p) {
    __shared__ __attribute__((aligned(16))) unsigned char smem[RING * 8192 + 64];
    const int tid = threadIdx.x, lane = tid & 63;
    const int wave = __builtin_amdgcn_readfirstlane(tid >> 6);
    const int b = blockIdx.x;
    const int g = (b & 7) * 8 + (b >> 5);
    const int mt = (b >> 3) & 3;
    const int xq = b & 7;
    const int gp = b >> 3;
    const int xd = xq ? 1 + (xq - 1 + gp) % 7 : 0;
    const unsigned lds0 = (unsigned)(size_t)(__attribute__((address_space(3))) unsigned char*)smem;
    const unsigned CTL = lds0 + RING * 8192;

    const unsigned long long t_start64 = __builtin_amdgcn_s_memrealtime();
    const unsigned t_start = (unsigned)t_start64;
    int rank = p.rank;
    gchar* myws = (gchar*)p.myws;
    const gchar* xin = (const gchar*)p.x;
    const gchar* win = (const gchar*)p.w;
    gchar* dws = (gchar*)p.ws[xd];
    if (wave == MW) asm volatile("ds_write_b32 %0, %1\n\ts_waitcnt lgkmcnt(0)" :: "v"(CTL + (lane & 15) * 4), "v"(0u) : "memory");
    asm volatile("s_barrier" ::: "memory");
    asm volatile("" : "+s"(xin), "+s"(win), "+s"(dws), "+s"(rank), "+s"(myws));
    gu32* const tmo = (gu32*)(myws + TMO_OFF);
    gu32* const cnt = (gu32*)(myws + CNT_OFF + (size_t)b * 128) + wave;
    const unsigned cnt_old = __hip_atomic_load(cnt, __ATOMIC_RELAXED, __HIP_MEMORY_SCOPE_AGENT);
    unsigned call;
    size_t slot;
#define GET_CALL() do { asm volatile("v_readfirstlane_b32 %0, %1\n\ts_nop 3" : "=s"(call) : "v"(cnt_old)); call += 1u; \
        slot = (size_t)(call & 1u) * SLOT_BYTES; } while (0)
#define PUT_CALL() do { if (lane == 0) __hip_atomic_store(cnt, call, __ATOMIC_RELAXED, __HIP_MEMORY_SCOPE_AGENT); } while (0)
    bool ok = true;

    if (__builtin_expect(wave < 2, 1)) {
        const unsigned dest = (unsigned)(rank + xd) & 7u;
        const gchar* src = xin + (size_t)(dest * MS + gp * 16 + (lane & 15)) * KS + wave * 256 + (lane >> 4) * 16;
        u32x4 v[4];
#pragma unroll
        for (int i = 0; i < 4; ++i) v[i] = *(const GAS u32x4*)(src + i * 64);
        GET_CALL();
        gchar* dst = dws + slot + XST_OFF + (size_t)(gp * NR + rank) * XSS + wave * 4096 + lane * 16;
#pragma unroll
        for (int i = 0; i < 4; ++i) {
            store16_sys(dst + i * 1024, v[i]);
            if (i == 0 && wave == 0) { lds_st(CTL + 12, (unsigned)__builtin_amdgcn_s_memrealtime() | 1u); STAMP(4); }
            wait_vm<0>();
        }
        if (wave == 0) STAMP(5);
        if (lds_ticket(CTL + 8, lane) == 1u && lane == 0)
            __hip_atomic_store((gu32*)(dws + FX_OFF + (gp >> 1) * 128) + (gp & 1) * 8 + rank, call,
                               __ATOMIC_RELAXED, __HIP_MEMORY_SCOPE_SYSTEM);
        PUT_CALL();
    } else if (wave < 4) {
        if (SDELAY && xq) __builtin_amdgcn_s_sleep(SDELAY);
        const int cg = lane & 15, sg = gp * 2 + ((lane >> 4) & 1);
        const int r = xq * 4 + (wave - 2) * 2 + (lane >> 5);
        const gchar* src = win + (size_t)(r * 16) * NN + sg * 64 + cg * 4;
        unsigned d[16];
#pragma unroll
        for (int i = 0; i < 16; ++i) d[i] = *(const gu32*)(src + (size_t)i * NN);
        GET_CALL();
        u32x4 o[4];
#pragma unroll
        for (int q = 0; q < 4; ++q) {
            const unsigned t0 = __builtin_amdgcn_perm(d[4 * q + 1], d[4 * q], 0x05010400u);
            const unsigned t1 = __builtin_amdgcn_perm(d[4 * q + 1], d[4 * q], 0x07030602u);
            const unsigned t2 = __builtin_amdgcn_perm(d[4 * q + 3], d[4 * q + 2], 0x05010400u);
            const unsigned t3 = __builtin_amdgcn_perm(d[4 * q + 3], d[4 * q + 2], 0x07030602u);
            o[0][q] = __builtin_amdgcn_perm(t2, t0, 0x05040100u);
            o[1][q] = __builtin_amdgcn_perm(t2, t0, 0x07060302u);
            o[2][q] = __builtin_amdgcn_perm(t3, t1, 0x05040100u);
            o[3][q] = __builtin_amdgcn_perm(t3, t1, 0x07060302u);
        }
        gchar* dst = myws + slot + (size_t)sg * GS + r * 1024 + cg * 16;
#pragma unroll
        for (int j = 0; j < 4; ++j) store16_sys(dst + j * 256, o[j]);
        if (wave == 2) STAMP(1);
        wait_vm<0>();
        if (wave == 2) STAMP(2);
        gu32* const ac = (gu32*)(myws + AC_OFF + gp * 128);
        if (lds_ticket(CTL + 4, lane) == 1u) {
            if (xq) {
                if (lane == 0) __hip_atomic_fetch_add(ac, 1u, __ATOMIC_RELAXED, __HIP_MEMORY_SCOPE_AGENT);
            } else {
                gchar* peer_l = (gchar*)p.ws[lane & 7];
                unsigned tx;
                while ((tx = __builtin_amdgcn_readfirstlane(lds_ld(CTL + 12))) == 0u) __builtin_amdgcn_s_sleep(1);
                wait_until(tx - 1u, TF);
                if (lane < NR)
                    __hip_atomic_store((gu32*)(peer_l + FW_OFF + (gp * 8 + rank) * 128), call,
                                       __ATOMIC_RELAXED, __HIP_MEMORY_SCOPE_SYSTEM);
                STAMP(12);
                ok = wait_flags<2>(ac, call * 7u, call, tmo);
                if (ok && lane < NR)
                    __hip_atomic_store((gu32*)(peer_l + FA_OFF + gp * 128) + rank, call,
                                       __ATOMIC_RELAXED, __HIP_MEMORY_SCOPE_SYSTEM);
                STAMP(13);
            }
        }
        PUT_CALL();
    } else if (wave > 4) {
        GET_CALL();
        PUT_CALL();
        if (wave == MW) {
            while (__builtin_amdgcn_readfirstlane(lds_ld(CTL + 20)) == 0u) __builtin_amdgcn_s_sleep(4);
            __builtin_amdgcn_s_sleep(32);
            ok = wait_flags<16>((const gu32*)(myws + FA_OFF + (g >> 1) * 128) + (lane & 7), call, call, tmo);
            lds_st(CTL + 16, ok ? 1u : 2u);
            STAMP(3);
        }
    } else {
        __builtin_amdgcn_s_setprio(3);
        const unsigned xcc = (unsigned)__builtin_amdgcn_s_getreg((3 << 11) | 20) & 0xFu;
        gu32* const iv = (gu32*)(myws + INV_OFF + xcc * 256);
        bool inv_done = false;
        if (b < 8) asm volatile("buffer_inv sc0 sc1" ::: "memory");
        if (b < 8) {
            wait_vm<0>();
            inv_done = true;
        }
        GET_CALL();
        if (b < 8 && lane == 0) __hip_atomic_store(iv, call, __ATOMIC_RELAXED, __HIP_MEMORY_SCOPE_AGENT);
        PUT_CALL();
        const gchar* base[NR];
        unsigned ldst[NR];
#pragma unroll
        for (int i = 0; i < NR; ++i) {
            const unsigned rel = (unsigned)(g + i) & 7u;
            base[i] = (const gchar*)p.ws[rel] + slot;
            ldst[i] = lds0 + ((rel + rank) & 7u) * 1024;
        }
        {
            const gu32* fw = lane < NR ? (const gu32*)(myws + FW_OFF + ((g >> 1) * 8 + lane) * 128) : iv;
            bool seen = lane > NR || (lane == NR && inv_done);
            wait_until(t_start, PT0);
            const unsigned long long t0 = __builtin_amdgcn_s_memrealtime();
            for (unsigned spin = 1;; ++spin) {
                if (!seen) seen = __hip_atomic_load(fw, __ATOMIC_RELAXED, __HIP_MEMORY_SCOPE_SYSTEM) >= call;
                const unsigned long long rdy = __ballot(seen);
                inv_done = inv_done || ((rdy >> NR) & 1);
                if ((rdy & 0xFFull) == 0xFFull && inv_done) break;
                if ((spin & 31u) == 0u) {
                    if (!inv_done) {
                        asm volatile("buffer_inv sc0 sc1\n\ts_waitcnt vmcnt(0)" ::: "memory");
                        inv_done = true;
                    }
                    if (give_up(tmo, call, t0)) { ok = false; break; }
                }
                __builtin_amdgcn_s_sleep(1);
            }
        }
        const unsigned long long t7 = __builtin_amdgcn_s_memrealtime();
        const unsigned tref = (unsigned)t7;
        lds_st(CTL + 20, 1u);
        unsigned voff = g * GS + lane * 16;
#pragma unroll 1
        for (int r = 0; r < ROUNDS && ok; ++r) {
            if (r == 4) {
                unsigned fa;
                while ((fa = __builtin_amdgcn_readfirstlane(lds_ld(CTL + 16))) == 0u) __builtin_amdgcn_s_sleep(1);
                if (fa != 1u) { ok = false; break; }
            }
            const unsigned sched = PACE * (unsigned)r;
            const unsigned now = (unsigned)__builtin_amdgcn_s_memrealtime() - tref;
            const bool behind = now > sched + LAG;
            if (now < sched) wait_until(tref, sched);
            if (r >= RING) {
                for (;;) {
                    const unsigned dn = lds_ld(CTL + 32 + (lane & (MW - 1)) * 4);
                    if (__all(dn + RING > (unsigned)r)) break;
                    wait_vm<0>();
                    lds_st(CTL, (unsigned)r);
                    __builtin_amdgcn_s_sleep(1);
                }
            }
            const unsigned so = (unsigned)(r % RING) * 8192;
            if (behind) {
#pragma unroll
                for (int i = 0; i < NR; ++i) {
                    glds16(base[i], voff, ldst[i] + so);
                    wait_vm<WIN - 1 + CATCH>();
                }
                wait_vm<NR>();
                lds_st(CTL, (unsigned)r);
            } else {
#pragma unroll
                for (int i = 0; i < NR; ++i) {
                    glds16(base[i], voff, ldst[i] + so);
                    wait_vm<WIN - 1>();
                    if (i == NR - 2) lds_st(CTL, (unsigned)r);
                }
            }
            voff += 1024;
        }
        wait_vm<0>();
        lds_st(CTL, ok ? (unsigned)ROUNDS : 0x80000000u);
        STAMP(8);
        return;
    }

    unsigned landed = 0;
    while ((landed = __builtin_amdgcn_readfirstlane(lds_ld(CTL))) < 2u) __builtin_amdgcn_s_sleep(8);
    const int mw = wave > 4 ? wave - 1 : wave;
    const int rt = mt * 8 + mw;
    ok = wait_flags<16>((const gu32*)(myws + FX_OFF + (rt >> 1) * 128) + (rt & 1) * 8 + (lane & 7), call, call, tmo) && ok;
    const gchar* abase = myws + slot + XST_OFF + (size_t)(rt * NR + (lane >> 4) * 2) * XSS + (lane & 15) * 16;
    constexpr int PF = 4;
    i32x4 a[PF][2];
#pragma unroll
    for (int u = 0; u < PF; ++u)
#pragma unroll
        for (int h = 0; h < 2; ++h) a[u][h] = *(const GAS i32x4*)(abase + h * XSS + u * 256);
    float sc = p.sx[0] * p.sw[0];
    float sce = -1.44269504088896341f * sc;
    GAS float* orow = (GAS float*)p.out + (size_t)(rt * 16 + (lane >> 4) * 4) * NN + g * 64 + (lane & 15) * 4;
    asm volatile("" : "+v"(sc), "+v"(sce), "+v"(orow));
    f32x4 acc[4];
#pragma unroll
    for (int j = 0; j < 4; ++j) acc[j] = f32x4{0.f, 0.f, 0.f, 0.f};
    const unsigned char* bl = smem + (lane >> 4) * 2048 + (lane & 15) * 16;
#pragma unroll 1
    for (int r0 = 0; r0 < ROUNDS; r0 += PF) {
        const bool more = r0 + PF < ROUNDS;
#pragma unroll
        for (int u = 0; u < PF; ++u) {
            const int r = r0 + u;
            while (landed <= (unsigned)r) {
                landed = __builtin_amdgcn_readfirstlane(lds_ld(CTL));
                if (landed <= (unsigned)r) __builtin_amdgcn_s_sleep(1);
            }
            const unsigned char* bs = bl + (r % RING) * 8192;
            i32x8 bf[4];
#pragma unroll
            for (int j = 0; j < 4; ++j) {
                const i32x4 lo = *(const i32x4*)(bs + j * 256);
                const i32x4 hi = *(const i32x4*)(bs + 1024 + j * 256);
                bf[j] = __builtin_shufflevector(lo, hi, 0, 1, 2, 3, 4, 5, 6, 7);
            }
            lds_st(CTL + 32 + mw * 4, (unsigned)r + 1u);
            __builtin_amdgcn_sched_barrier(0);
            const i32x8 af = __builtin_shufflevector(a[u][0], a[u][1], 0, 1, 2, 3, 4, 5, 6, 7);
#pragma unroll
            for (int j = 0; j < 4; ++j)
                acc[j] = __builtin_amdgcn_mfma_scale_f32_16x16x128_f8f6f4(af, bf[j], acc[j], 0, 0,
                                                                          0, 0x7F7F7F7F, 0, 0x7F7F7F7F);
            asm volatile("" : "+v"(acc[0]), "+v"(acc[1]), "+v"(acc[2]), "+v"(acc[3]));
            const int rn = more ? r + PF : r;
#pragma unroll
            for (int h = 0; h < 2; ++h) a[u][h] = *(const GAS i32x4*)(abase + h * XSS + rn * 256);
        }
    }
    if (wave == 0) STAMP(10);
    const bool dead = !ok || landed != (unsigned)ROUNDS;
    const float scy = dead ? __builtin_nanf("") : sc;
    constexpr float LIM = 60.f * 1.44269504088896341f;
#pragma unroll
    for (int i = 0; i < 4; ++i) {
        f32x4 o;
#pragma unroll
        for (int j = 0; j < 4; ++j) {
            const float e = __builtin_amdgcn_exp2f(__builtin_amdgcn_fmed3f(acc[j][i] * sce, -LIM, LIM));
            o[j] = (acc[j][i] * scy) * __builtin_amdgcn_rcpf(1.f + e);
        }
        asm volatile("global_store_dwordx4 %0, %1, off sc1\n\ts_nop 1" :: "v"(orow + (size_t)i * NN), "v"(o) : "memory");
    }
}

}

extern "C" void kernel_launch(int rank, int nranks, ncclComm_t comm, hipStream_t stream,
                              void* const* d_in, const int* in_sizes, int n_in,
                              void* d_out, int out_size, const int* in_dtypes, int out_dtype,
                              void* const* d_out_all, void* const* d_ws_all, size_t ws_bytes) {
    (void)comm; (void)in_dtypes; (void)out_dtype; (void)d_out_all;
    if (nranks != NR || (unsigned)rank >= (unsigned)NR || ws_bytes < WS_NEED || n_in != 4
        || in_sizes[0] != NR * MS * KS || in_sizes[1] != KS * NN || out_size != MS * NN) {
        fprintf(stderr, "unexpected configuration\n");
        return;
    }
    Params p;
    p.x = (const unsigned char*)d_in[0];
    p.w = (const unsigned char*)d_in[1];
    p.sx = (const float*)d_in[2];
    p.sw = (const float*)d_in[3];
    p.out = (float*)d_out;
    for (int i = 0; i < NR; ++i) p.ws[i] = (unsigned char*)d_ws_all[(rank + i) % NR];
    p.myws = p.ws[0];
    p.rank = rank;
    p.pad = 0;
    p.pad2 = nullptr;
    gemm_gather_silu<<<NWG, (MW + 1) * 64, 0, stream>>>(p);
    HIP_CHECK(hipGetLastError());
}
```

```cpp
#include <hip/hip_runtime.h>
#include <rccl/rccl.h>
#include <cstdio>

#define HIP_CHECK(expr) do { hipError_t _e = (expr); if (_e != hipSuccess) { \
    fprintf(stderr, "HIP error %s at %s:%d\n", hipGetErrorString(_e), __FILE__, __LINE__); \
    return; } } while (0)

namespace {

constexpr int NR = 8;
constexpr int MS = 512;
constexpr int KS = 512;
constexpr int NN = 4096;
constexpr int NWG = 256;
constexpr int MW = 8;
constexpr int ROUNDS = KS / 16;
constexpr int RING = 16;
constexpr int WIN = 8;
constexpr unsigned PACE = 107;
constexpr unsigned LAG = 90;
constexpr int CATCH = 4;
#define TFX 280
#define OSTX "sc1"
#define PRE0 8
#define SDELAY 8
#define PT0 380
constexpr unsigned TF = TFX;

constexpr int GS = 32768;
constexpr int XSS = 8192 + 256;
constexpr size_t WST_BYTES = (size_t)64 * GS;
constexpr size_t XST_BYTES = (size_t)32 * NR * XSS;
constexpr size_t XST_OFF = WST_BYTES;
constexpr size_t SLOT_BYTES = WST_BYTES + XST_BYTES;
constexpr size_t FW_OFF = 2 * SLOT_BYTES;
constexpr size_t FA_OFF = FW_OFF + 32 * 8 * 128;
constexpr size_t AC_OFF = FA_OFF + 32 * 128;
constexpr size_t FX_OFF = AC_OFF + 32 * 128;
constexpr size_t CNT_OFF = FX_OFF + 16 * 128;
constexpr size_t TMO_OFF = CNT_OFF + NWG * 128;
constexpr size_t INV_OFF = TMO_OFF + 128;
constexpr size_t WS_NEED = INV_OFF + 16 * 256;
constexpr unsigned long long DEADLINE_TICKS = 50000000ull;

#define GAS __attribute__((address_space(1)))
typedef GAS unsigned char gchar;
typedef GAS unsigned gu32;
typedef unsigned int u32x4 __attribute__((ext_vector_type(4)));
typedef int i32x8 __attribute__((ext_vector_type(8)));
typedef int i32x4 __attribute__((ext_vector_type(4)));
typedef float f32x4 __attribute__((ext_vector_type(4)));

#define STAMP(i) do {} while (0)

struct Params {
    const unsigned char* x;
    const unsigned char* w;
    unsigned char* myws;
    int rank;
    int pad;
    const float* sx;
    const float* sw;
    float* out;
    unsigned char* pad2;
    unsigned char* ws[NR];
};

__device__ __forceinline__ void store16_sys(gchar* p, u32x4 v) {
    asm volatile("global_store_dwordx4 %0, %1, off sc0 sc1\n\ts_nop 1" :: "v"(p), "v"(v) : "memory");
}
__device__ __forceinline__ void lds_st(unsigned addr, unsigned v) {
    asm volatile("ds_write_b32 %0, %1" :: "v"(addr), "v"(v) : "memory");
}
__device__ __forceinline__ unsigned lds_ld(unsigned addr) {
    unsigned v;
    asm volatile("ds_read_b32 %0, %1\n\ts_waitcnt lgkmcnt(0)" : "=v"(v) : "v"(addr) : "memory");
    return v;
}
__device__ __forceinline__ unsigned lds_ticket(unsigned addr, int lane) {
    unsigned old;
    asm volatile("ds_add_rtn_u32 %0, %1, %2\n\ts_waitcnt lgkmcnt(0)"
                 : "=v"(old) : "v"(addr), "v"(lane == 0 ? 1u : 0u) : "memory");
    return __builtin_amdgcn_readfirstlane(old);
}
__device__ __forceinline__ void wait_until(unsigned t0, unsigned ticks) {
    while ((unsigned)__builtin_amdgcn_s_memrealtime() - t0 < ticks) __builtin_amdgcn_s_sleep(1);
}
__device__ __forceinline__ void glds16(const gchar* sbase, unsigned voff, unsigned lds_dst) {
    unsigned keep;
    asm volatile("s_mov_b32 %0, m0\n\ts_mov_b32 m0, %3\n\ts_nop 0\n\tglobal_load_lds_dwordx4 %1, %2\n\ts_mov_b32 m0, %0"
                 : "=&s"(keep) : "v"(voff), "s"(sbase), "s"(lds_dst) : "memory");
}
template <int N> __device__ __forceinline__ void wait_vm() { asm volatile("s_waitcnt vmcnt(%0)" :: "i"(N) : "memory"); }

__device__ __forceinline__ bool give_up(gu32* tmo, unsigned call, unsigned long long t0) {
    if (__hip_atomic_load(tmo, __ATOMIC_RELAXED, __HIP_MEMORY_SCOPE_AGENT) == call
        || __builtin_amdgcn_s_memrealtime() - t0 > DEADLINE_TICKS) {
        __hip_atomic_store(tmo, call, __ATOMIC_RELAXED, __HIP_MEMORY_SCOPE_AGENT);
        return true;
    }
    return false;
}
template <int SLEEP>
__device__ __forceinline__ bool wait_flags(const gu32* flag, unsigned want, unsigned call, gu32* tmo) {
    const unsigned long long t0 = __builtin_amdgcn_s_memrealtime();
    for (unsigned spin = 1;; ++spin) {
        const unsigned v = __hip_atomic_load(flag, __ATOMIC_RELAXED, __HIP_MEMORY_SCOPE_SYSTEM);
        if (__all(v >= want)) break;
        if ((spin & 63u) == 0u && give_up(tmo, call, t0)) return false;
        __builtin_amdgcn_s_sleep(SLEEP);
    }
    asm volatile("" ::: "memory");
    return true;
}

__global__ void __launch_bounds__((MW + 1) * 64, 1) gemm_gather_silu(Params p) {
    __shared__ __attribute__((aligned(16))) unsigned char smem[RING * 8192 + 64];
    const int tid = threadIdx.x, lane = tid & 63;
    const int wave = __builtin_amdgcn_readfirstlane(tid >> 6);
    const int b = blockIdx.x;
    const int g = (b & 7) * 8 + (b >> 5);
    const int mt = (b >> 3) & 3;
    const int xq = b & 7;
    const int gp = b >> 3;
    const int xd = xq ? 1 + (xq - 1 + gp) % 7 : 0;
    const unsigned lds0 = (unsigned)(size_t)(__attribute__((address_space(3))) unsigned char*)smem;
    const unsigned CTL = lds0 + RING * 8192;

    const unsigned long long t_start64 = __builtin_amdgcn_s_memrealtime();
    const unsigned t_start = (unsigned)t_start64;
    int rank = p.rank;
    gchar* myws = (gchar*)p.myws;
    const gchar* xin = (const gchar*)p.x;
    const gchar* win = (const gchar*)p.w;
    gchar* dws = (gchar*)p.ws[xd];
    if (wave == MW) asm volatile("ds_write_b32 %0, %1\n\ts_waitcnt lgkmcnt(0)" :: "v"(CTL + (lane & 15) * 4), "v"(0u) : "memory");
    asm volatile("s_barrier" ::: "memory");
    asm volatile("" : "+s"(xin), "+s"(win), "+s"(dws), "+s"(rank), "+s"(myws));
    gu32* const tmo = (gu32*)(myws + TMO_OFF);
    gu32* const cnt = (gu32*)(myws + CNT_OFF + (size_t)b * 128) + wave;
    const unsigned cnt_old = __hip_atomic_load(cnt, __ATOMIC_RELAXED, __HIP_MEMORY_SCOPE_AGENT);
    unsigned call;
    size_t slot;
#define GET_CALL() do { asm volatile("v_readfirstlane_b32 %0, %1\n\ts_nop 3" : "=s"(call) : "v"(cnt_old)); call += 1u; \
        slot = (size_t)(call & 1u) * SLOT_BYTES; } while (0)
#define PUT_CALL() do { if (lane == 0) __hip_atomic_store(cnt, call, __ATOMIC_RELAXED, __HIP_MEMORY_SCOPE_AGENT); } while (0)
    bool ok = true;

    if (__builtin_expect(wave < 2, 1)) {
        const unsigned dest = (unsigned)(rank + xd) & 7u;
        const gchar* src = xin + (size_t)(dest * MS + gp * 16 + (lane & 15)) * KS + wave * 256 + (lane >> 4) * 16;
        u32x4 v[4];
#pragma unroll
        for (int i = 0; i < 4; ++i) v[i] = *(const GAS u32x4*)(src + i * 64);
        GET_CALL();
        gchar* dst = dws + slot + XST_OFF + (size_t)(gp * NR + rank) * XSS + wave * 4096 + lane * 16;
#pragma unroll
        for (int i = 0; i < 4; ++i) {
            store16_sys(dst + i * 1024, v[i]);
            if (i == 0 && wave == 0) { lds_st(CTL + 12, (unsigned)__builtin_amdgcn_s_memrealtime() | 1u); STAMP(4); }
            wait_vm<0>();
        }
        if (wave == 0) STAMP(5);
        if (lds_ticket(CTL + 8, lane) == 1u && lane == 0)
            __hip_atomic_store((gu32*)(dws + FX_OFF + (gp >> 1) * 128) + (gp & 1) * 8 + rank, call,
                               __ATOMIC_RELAXED, __HIP_MEMORY_SCOPE_SYSTEM);
        PUT_CALL();
    } else if (wave < 4) {
        if (SDELAY && xq) __builtin_amdgcn_s_sleep(SDELAY);
        const int cg = lane & 15, sg = gp * 2 + ((lane >> 4) & 1);
        const int r = xq * 4 + (wave - 2) * 2 + (lane >> 5);
        const gchar* src = win + (size_t)(r * 16) * NN + sg * 64 + cg * 4;
        unsigned d[16];
#pragma unroll
        for (int i = 0; i < 16; ++i) d[i] = *(const gu32*)(src + (size_t)i * NN);
        GET_CALL();
        u32x4 o[4];
#pragma unroll
        for (int q = 0; q < 4; ++q) {
            const unsigned t0 = __builtin_amdgcn_perm(d[4 * q + 1], d[4 * q], 0x05010400u);
            const unsigned t1 = __builtin_amdgcn_perm(d[4 * q + 1], d[4 * q], 0x07030602u);
            const unsigned t2 = __builtin_amdgcn_perm(d[4 * q + 3], d[4 * q + 2], 0x05010400u);
            const unsigned t3 = __builtin_amdgcn_perm(d[4 * q + 3], d[4 * q + 2], 0x07030602u);
            o[0][q] = __builtin_amdgcn_perm(t2, t0, 0x05040100u);
            o[1][q] = __builtin_amdgcn_perm(t2, t0, 0x07060302u);
            o[2][q] = __builtin_amdgcn_perm(t3, t1, 0x05040100u);
            o[3][q] = __builtin_amdgcn_perm(t3, t1, 0x07060302u);
        }
        gchar* dst = myws + slot + (size_t)sg * GS + r * 1024 + cg * 16;
#pragma unroll
        for (int j = 0; j < 4; ++j) store16_sys(dst + j * 256, o[j]);
        if (wave == 2) STAMP(1);
        wait_vm<0>();
        if (wave == 2) STAMP(2);
        gu32* const ac = (gu32*)(myws + AC_OFF + gp * 128);
        if (lds_ticket(CTL + 4, lane) == 1u) {
            if (xq) {
                if (lane == 0) __hip_atomic_fetch_add(ac, 1u, __ATOMIC_RELAXED, __HIP_MEMORY_SCOPE_AGENT);
            } else {
                gchar* peer_l = (gchar*)p.ws[lane & 7];
                unsigned tx;
                while ((tx = __builtin_amdgcn_readfirstlane(lds_ld(CTL + 12))) == 0u) __builtin_amdgcn_s_sleep(1);
                wait_until(tx - 1u, TF);
                if (lane < NR)
                    __hip_atomic_store((gu32*)(peer_l + FW_OFF + (gp * 8 + rank) * 128), call,
                                       __ATOMIC_RELAXED, __HIP_MEMORY_SCOPE_SYSTEM);
                STAMP(12);
                ok = wait_flags<2>(ac, call * 7u, call, tmo);
                if (ok && lane < NR)
                    __hip_atomic_store((gu32*)(peer_l + FA_OFF + gp * 128) + rank, call,
                                       __ATOMIC_RELAXED, __HIP_MEMORY_SCOPE_SYSTEM);
                STAMP(13);
            }
        }
        PUT_CALL();
    } else if (wave > 4) {
        GET_CALL();
        PUT_CALL();
        if (wave == MW) {
            while (__builtin_amdgcn_readfirstlane(lds_ld(CTL + 20)) == 0u) __builtin_amdgcn_s_sleep(4);
            __builtin_amdgcn_s_sleep(32);
            ok = wait_flags<16>((const gu32*)(myws + FA_OFF + (g >> 1) * 128) + (lane & 7), call, call, tmo);
            lds_st(CTL + 16, ok ? 1u : 2u);
            STAMP(3);
        }
    } else {
        __builtin_amdgcn_s_setprio(3);
        const unsigned xcc = (unsigned)__builtin_amdgcn_s_getreg((3 << 11) | 20) & 0xFu;
        gu32* const iv = (gu32*)(myws + INV_OFF + xcc * 256);
        bool inv_done = false;
        if (b < 8) asm volatile("buffer_inv sc0 sc1" ::: "memory");
        if (b < 8) {
            wait_vm<0>();
            inv_done = true;
        }
        GET_CALL();
        if (b < 8 && lane == 0) __hip_atomic_store(iv, call, __ATOMIC_RELAXED, __HIP_MEMORY_SCOPE_AGENT);
        PUT_CALL();
        const gchar* base[NR];
        unsigned ldst[NR];
#pragma unroll
        for (int i = 0; i < NR; ++i) {
            const unsigned rel = (unsigned)(g + i) & 7u;
            base[i] = (const gchar*)p.ws[rel] + slot;
            ldst[i] = lds0 + ((rel + rank) & 7u) * 1024;
        }
        {
            const gu32* fw = lane < NR ? (const gu32*)(myws + FW_OFF + ((g >> 1) * 8 + lane) * 128) : iv;
            bool seen = lane > NR || (lane == NR && inv_done);
            wait_until(t_start, PT0);
            const unsigned long long t0 = __builtin_amdgcn_s_memrealtime();
            for (unsigned spin = 1;; ++spin) {
                if (!seen) seen = __hip_atomic_load(fw, __ATOMIC_RELAXED, __HIP_MEMORY_SCOPE_SYSTEM) >= call;
                const unsigned long long rdy = __ballot(seen);
                inv_done = inv_done || ((rdy >> NR) & 1);
                if ((rdy & 0xFFull) == 0xFFull && inv_done) break;
                if ((spin & 31u) == 0u) {
                    if (!inv_done) {
                        asm volatile("buffer_inv sc0 sc1\n\ts_waitcnt vmcnt(0)" ::: "memory");
                        inv_done = true;
                    }
                    if (give_up(tmo, call, t0)) { ok = false; break; }
                }
                __builtin_amdgcn_s_sleep(1);
            }
        }
        const unsigned long long t7 = __builtin_amdgcn_s_memrealtime();
        const unsigned tref = (unsigned)t7;
        lds_st(CTL + 20, 1u);
        unsigned voff = g * GS + lane * 16;
#pragma unroll 1
        for (int r = 0; r < ROUNDS && ok; ++r) {
            if (r == 4) {
                unsigned fa;
                while ((fa = __builtin_amdgcn_readfirstlane(lds_ld(CTL + 16))) == 0u) __builtin_amdgcn_s_sleep(1);
                if (fa != 1u) { ok = false; break; }
            }
            const unsigned sched = PACE * (unsigned)r;
            const unsigned now = (unsigned)__builtin_amdgcn_s_memrealtime() - tref;
            const bool behind = now > sched + LAG;
            if (now < sched) wait_until(tref, sched);
            if (r >= RING) {
                for (;;) {
                    const unsigned dn = lds_ld(CTL + 32 + (lane & (MW - 1)) * 4);
                    if (__all(dn + RING > (unsigned)r)) break;
                    wait_vm<0>();
                    lds_st(CTL, (unsigned)r);
                    __builtin_amdgcn_s_sleep(1);
                }
            }
            const unsigned so = (unsigned)(r % RING) * 8192;
            if (behind) {
#pragma unroll
                for (int i = 0; i < NR; ++i) {
                    glds16(base[i], voff, ldst[i] + so);
                    wait_vm<WIN - 1 + CATCH>();
                }
                wait_vm<NR>();
                lds_st(CTL, (unsigned)r);
            } else {
#pragma unroll
                for (int i = 0; i < NR; ++i) {
                    glds16(base[i], voff, ldst[i] + so);
                    wait_vm<WIN - 1>();
                    if (i == NR - 2) lds_st(CTL, (unsigned)r);
                }
            }
            voff += 1024;
        }
        wait_vm<0>();
        lds_st(CTL, ok ? (unsigned)ROUNDS : 0x80000000u);
        STAMP(8);
        return;
    }

    unsigned landed = 0;
    while ((landed = __builtin_amdgcn_readfirstlane(lds_ld(CTL))) < 2u) __builtin_amdgcn_s_sleep(8);
    const int mw = wave > 4 ? wave - 1 : wave;
    const int rt = mt * 8 + mw;
    ok = wait_flags<16>((const gu32*)(myws + FX_OFF + (rt >> 1) * 128) + (rt & 1) * 8 + (lane & 7), call, call, tmo) && ok;
    const gchar* abase = myws + slot + XST_OFF + (size_t)(rt * NR + (lane >> 4) * 2) * XSS + (lane & 15) * 16;
    constexpr int PF = 4;
    i32x4 a[PF][2];
#pragma unroll
    for (int u = 0; u < PF; ++u)
#pragma unroll
        for (int h = 0; h < 2; ++h) a[u][h] = *(const GAS i32x4*)(abase + h * XSS + u * 256);
    float sc = p.sx[0] * p.sw[0];
    float sce = -1.44269504088896341f * sc;
    GAS float* orow = (GAS float*)p.out + (size_t)(rt * 16 + (lane >> 4) * 4) * NN + g * 64 + (lane & 15) * 4;
    asm volatile("" : "+v"(sc), "+v"(sce), "+v"(orow));
    f32x4 acc[4];
#pragma unroll
    for (int j = 0; j < 4; ++j) acc[j] = f32x4{0.f, 0.f, 0.f, 0.f};
    const unsigned char* bl = smem + (lane >> 4) * 2048 + (lane & 15) * 16;
#pragma unroll 1
    for (int r0 = 0; r0 < ROUNDS; r0 += PF) {
        const bool more = r0 + PF < ROUNDS;
#pragma unroll
        for (int u = 0; u < PF; ++u) {
            const int r = r0 + u;
            while (landed <= (unsigned)r) {
                landed = __builtin_amdgcn_readfirstlane(lds_ld(CTL));
                if (landed <= (unsigned)r) __builtin_amdgcn_s_sleep(1);
            }
            const unsigned char* bs = bl + (r % RING) * 8192;
            i32x8 bf[4];
#pragma unroll
            for (int j = 0; j < 4; ++j) {
                const i32x4 lo = *(const i32x4*)(bs + j * 256);
                const i32x4 hi = *(const i32x4*)(bs + 1024 + j * 256);
                bf[j] = __builtin_shufflevector(lo, hi, 0, 1, 2, 3, 4, 5, 6, 7);
            }
            lds_st(CTL + 32 + mw * 4, (unsigned)r + 1u);
            __builtin_amdgcn_sched_barrier(0);
            const i32x8 af = __builtin_shufflevector(a[u][0], a[u][1], 0, 1, 2, 3, 4, 5, 6, 7);
#pragma unroll
            for (int j = 0; j < 4; ++j)
                acc[j] = __builtin_amdgcn_mfma_scale_f32_16x16x128_f8f6f4(af, bf[j], acc[j], 0, 0,
                                                                          0, 0x7F7F7F7F, 0, 0x7F7F7F7F);
            asm volatile("" : "+v"(acc[0]), "+v"(acc[1]), "+v"(acc[2]), "+v"(acc[3]));
            if (r0 >= PRE0 && r0 < PRE0 + 16 && u == (mw & 3)) {
                const f32x4 z = {__builtin_nanf(""), __builtin_nanf(""), __builtin_nanf(""), __builtin_nanf("")};
                asm volatile("global_store_dwordx4 %0, %1, off sc1\n\ts_nop 1"
                             :: "v"(orow + (size_t)((r0 - PRE0) / PF) * NN), "v"(z) : "memory");
            }
            const int rn = more ? r + PF : r;
#pragma unroll
            for (int h = 0; h < 2; ++h) a[u][h] = *(const GAS i32x4*)(abase + h * XSS + rn * 256);
        }
    }
    if (wave == 0) STAMP(10);
    const bool dead = !ok || landed != (unsigned)ROUNDS;
    const float scy = dead ? __builtin_nanf("") : sc;
    constexpr float LIM = 60.f * 1.44269504088896341f;
#pragma unroll
    for (int i = 0; i < 4; ++i) {
        f32x4 o;
#pragma unroll
        for (int j = 0; j < 4; ++j) {
            const float e = __builtin_amdgcn_exp2f(__builtin_amdgcn_fmed3f(acc[j][i] * sce, -LIM, LIM));
            o[j] = (acc[j][i] * scy) * __builtin_amdgcn_rcpf(1.f + e);
        }
        asm volatile("global_store_dwordx4 %0, %1, off " OSTX "\n\ts_nop 1" :: "v"(orow + (size_t)i * NN), "v"(o) : "memory");
    }
}

}

extern "C" void kernel_launch(int rank, int nranks, ncclComm_t comm, hipStream_t stream,
                              void* const* d_in, const int* in_sizes, int n_in,
                              void* d_out, int out_size, const int* in_dtypes, int out_dtype,
                              void* const* d_out_all, void* const* d_ws_all, size_t ws_bytes) {
    (void)comm; (void)in_dtypes; (void)out_dtype; (void)d_out_all;
    if (nranks != NR || (unsigned)rank >= (unsigned)NR || ws_bytes < WS_NEED || n_in != 4
        || in_sizes[0] != NR * MS * KS || in_sizes[1] != KS * NN || out_size != MS * NN) {
        fprintf(stderr, "unexpected configuration\n");
        return;
    }
    Params p;
    p.x = (const unsigned char*)d_in[0];
    p.w = (const unsigned char*)d_in[1];
    p.sx = (const float*)d_in[2];
    p.sw = (const float*)d_in[3];
    p.out = (float*)d_out;
    for (int i = 0; i < NR; ++i) p.ws[i] = (unsigned char*)d_ws_all[(rank + i) % NR];
    p.myws = p.ws[0];
    p.rank = rank;
    p.pad = 0;
    p.pad2 = nullptr;
    gemm_gather_silu<<<NWG, (MW + 1) * 64, 0, stream>>>(p);
    HIP_CHECK(hipGetLastError());
}
```

```cpp
#include <hip/hip_runtime.h>
#include <rccl/rccl.h>
#include <cstdio>

#define HIP_CHECK(expr) do { hipError_t _e = (expr); if (_e != hipSuccess) { \
    fprintf(stderr, "HIP error %s at %s:%d\n", hipGetErrorString(_e), __FILE__, __LINE__); \
    return; } } while (0)

namespace {

constexpr int NR = 8;
constexpr int MS = 512;
constexpr int KS = 512;
constexpr int NN = 4096;
constexpr int NWG = 256;
constexpr int MW = 8;
constexpr int ROUNDS = KS / 16;
constexpr int RING = 16;
constexpr int WIN = 8;
constexpr unsigned PACE = 107;
constexpr unsigned LAG = 90;
constexpr int CATCH = 4;
constexpr unsigned TF = 280;
constexpr unsigned PT0 = 380;
constexpr int PRE0 = 8;

constexpr int GS = 32768;
constexpr int XSS = 8192 + 256;
constexpr size_t WST_BYTES = (size_t)64 * GS;
constexpr size_t XST_BYTES = (size_t)32 * NR * XSS;
constexpr size_t XST_OFF = WST_BYTES;
constexpr size_t SLOT_BYTES = WST_BYTES + XST_BYTES;
constexpr size_t FW_OFF = 2 * SLOT_BYTES;
constexpr size_t FA_OFF = FW_OFF + 32 * 8 * 128;
constexpr size_t AC_OFF = FA_OFF + 32 * 128;
constexpr size_t FX_OFF = AC_OFF + 32 * 128;
constexpr size_t CNT_OFF = FX_OFF + 16 * 128;
constexpr size_t TMO_OFF = CNT_OFF + NWG * 128;
constexpr size_t INV_OFF = TMO_OFF + 128;
constexpr size_t WS_NEED = INV_OFF + 16 * 256;
constexpr unsigned long long DEADLINE_TICKS = 100000000ull;

#define GAS __attribute__((address_space(1)))
typedef GAS unsigned char gchar;
typedef GAS unsigned gu32;
typedef unsigned int u32x4 __attribute__((ext_vector_type(4)));
typedef int i32x8 __attribute__((ext_vector_type(8)));
typedef int i32x4 __attribute__((ext_vector_type(4)));
typedef float f32x4 __attribute__((ext_vector_type(4)));

struct Params {
    const unsigned char* x;
    const unsigned char* w;
    unsigned char* myws;
    int rank;
    int pad;
    const float* sx;
    const float* sw;
    float* out;
    unsigned char* pad2;
    unsigned char* ws[NR];
};

__device__ __forceinline__ void store16_sys(gchar* p, u32x4 v) {
    asm volatile("global_store_dwordx4 %0, %1, off sc0 sc1\n\ts_nop 1" :: "v"(p), "v"(v) : "memory");
}
__device__ __forceinline__ void lds_st(unsigned addr, unsigned v) {
    asm volatile("ds_write_b32 %0, %1" :: "v"(addr), "v"(v) : "memory");
}
__device__ __forceinline__ unsigned lds_ld(unsigned addr) {
    unsigned v;
    asm volatile("ds_read_b32 %0, %1\n\ts_waitcnt lgkmcnt(0)" : "=v"(v) : "v"(addr) : "memory");
    return v;
}
__device__ __forceinline__ unsigned lds_ticket(unsigned addr, int lane) {
    unsigned old;
    asm volatile("ds_add_rtn_u32 %0, %1, %2\n\ts_waitcnt lgkmcnt(0)"
                 : "=v"(old) : "v"(addr), "v"(lane == 0 ? 1u : 0u) : "memory");
    return __builtin_amdgcn_readfirstlane(old);
}
__device__ __forceinline__ void wait_until(unsigned t0, unsigned ticks) {
    while ((unsigned)__builtin_amdgcn_s_memrealtime() - t0 < ticks) __builtin_amdgcn_s_sleep(1);
}
__device__ __forceinline__ void glds16(const gchar* sbase, unsigned voff, unsigned lds_dst) {
    unsigned keep;
    asm volatile("s_mov_b32 %0, m0\n\ts_mov_b32 m0, %3\n\ts_nop 0\n\tglobal_load_lds_dwordx4 %1, %2\n\ts_mov_b32 m0, %0"
                 : "=&s"(keep) : "v"(voff), "s"(sbase), "s"(lds_dst) : "memory");
}
template <int N> __device__ __forceinline__ void wait_vm() { asm volatile("s_waitcnt vmcnt(%0)" :: "i"(N) : "memory"); }

__device__ __forceinline__ bool give_up(gu32* tmo, unsigned call, unsigned long long t0) {
    if (__hip_atomic_load(tmo, __ATOMIC_RELAXED, __HIP_MEMORY_SCOPE_AGENT) == call
        || __builtin_amdgcn_s_memrealtime() - t0 > DEADLINE_TICKS) {
        __hip_atomic_store(tmo, call, __ATOMIC_RELAXED, __HIP_MEMORY_SCOPE_AGENT);
        return true;
    }
    return false;
}
template <int SLEEP>
__device__ __forceinline__ bool wait_flags(const gu32* flag, unsigned want, unsigned call, gu32* tmo) {
    const unsigned long long t0 = __builtin_amdgcn_s_memrealtime();
    for (unsigned spin = 1;; ++spin) {
        const unsigned v = __hip_atomic_load(flag, __ATOMIC_RELAXED, __HIP_MEMORY_SCOPE_SYSTEM);
        if (__all(v >= want)) break;
        if ((spin & 63u) == 0u && give_up(tmo, call, t0)) return false;
        __builtin_amdgcn_s_sleep(SLEEP);
    }
    asm volatile("" ::: "memory");
    return true;
}

__global__ void __launch_bounds__((MW + 1) * 64, 1) gemm_gather_silu(Params p) {
    __shared__ __attribute__((aligned(16))) unsigned char smem[RING * 8192 + 64];
    const int tid = threadIdx.x, lane = tid & 63;
    const int wave = __builtin_amdgcn_readfirstlane(tid >> 6);
    const int b = blockIdx.x;
    const int g = (b & 7) * 8 + (b >> 5);
    const int mt = (b >> 3) & 3;
    const int xq = b & 7;
    const int gp = b >> 3;
    const int xd = xq ? 1 + (xq - 1 + gp) % 7 : 0;
    const unsigned lds0 = (unsigned)(size_t)(__attribute__((address_space(3))) unsigned char*)smem;
    const unsigned CTL = lds0 + RING * 8192;

    const unsigned t_start = (unsigned)__builtin_amdgcn_s_memrealtime();
    int rank = p.rank;
    gchar* myws = (gchar*)p.myws;
    const gchar* xin = (const gchar*)p.x;
    const gchar* win = (const gchar*)p.w;
    gchar* dws = (gchar*)p.ws[xd];
    if (wave == MW) asm volatile("ds_write_b32 %0, %1\n\ts_waitcnt lgkmcnt(0)" :: "v"(CTL + (lane & 15) * 4), "v"(0u) : "memory");
    asm volatile("s_barrier" ::: "memory");
    asm volatile("" : "+s"(xin), "+s"(win), "+s"(dws), "+s"(rank), "+s"(myws));
    gu32* const tmo = (gu32*)(myws + TMO_OFF);
    gu32* const cnt = (gu32*)(myws + CNT_OFF + (size_t)b * 128) + wave;
    const unsigned cnt_old = __hip_atomic_load(cnt, __ATOMIC_RELAXED, __HIP_MEMORY_SCOPE_AGENT);
    unsigned call;
    size_t slot;
#define GET_CALL() do { asm volatile("v_readfirstlane_b32 %0, %1\n\ts_nop 3" : "=s"(call) : "v"(cnt_old)); call += 1u; \
        slot = (size_t)(call & 1u) * SLOT_BYTES; } while (0)
#define PUT_CALL() do { if (lane == 0) __hip_atomic_store(cnt, call, __ATOMIC_RELAXED, __HIP_MEMORY_SCOPE_AGENT); } while (0)
    bool ok = true;

    if (__builtin_expect(wave < 2, 1)) {
        const unsigned dest = (unsigned)(rank + xd) & 7u;
        const gchar* src = xin + (size_t)(dest * MS + gp * 16 + (lane & 15)) * KS + wave * 256 + (lane >> 4) * 16;
        u32x4 v[4];
#pragma unroll
        for (int i = 0; i < 4; ++i) v[i] = *(const GAS u32x4*)(src + i * 64);
        GET_CALL();
        gchar* dst = dws + slot + XST_OFF + (size_t)(gp * NR + rank) * XSS + wave * 4096 + lane * 16;
#pragma unroll
        for (int i = 0; i < 4; ++i) {
            store16_sys(dst + i * 1024, v[i]);
            if (i == 0 && wave == 0) lds_st(CTL + 12, (unsigned)__builtin_amdgcn_s_memrealtime() | 1u);
            wait_vm<0>();
        }
        if (lds_ticket(CTL + 8, lane) == 1u && lane == 0)
            __hip_atomic_store((gu32*)(dws + FX_OFF + (gp >> 1) * 128) + (gp & 1) * 8 + rank, call,
                               __ATOMIC_RELAXED, __HIP_MEMORY_SCOPE_SYSTEM);
        PUT_CALL();
    } else if (wave < 4) {
        if (xq) __builtin_amdgcn_s_sleep(8);
        const int cg = lane & 15, sg = gp * 2 + ((lane >> 4) & 1);
        const int r = xq * 4 + (wave - 2) * 2 + (lane >> 5);
        const gchar* src = win + (size_t)(r * 16) * NN + sg * 64 + cg * 4;
        unsigned d[16];
#pragma unroll
        for (int i = 0; i < 16; ++i) d[i] = *(const gu32*)(src + (size_t)i * NN);
        GET_CALL();
        u32x4 o[4];
#pragma unroll
        for (int q = 0; q < 4; ++q) {
            const unsigned t0 = __builtin_amdgcn_perm(d[4 * q + 1], d[4 * q], 0x05010400u);
            const unsigned t1 = __builtin_amdgcn_perm(d[4 * q + 1], d[4 * q], 0x07030602u);
            const unsigned t2 = __builtin_amdgcn_perm(d[4 * q + 3], d[4 * q + 2], 0x05010400u);
            const unsigned t3 = __builtin_amdgcn_perm(d[4 * q + 3], d[4 * q + 2], 0x07030602u);
            o[0][q] = __builtin_amdgcn_perm(t2, t0, 0x05040100u);
            o[1][q] = __builtin_amdgcn_perm(t2, t0, 0x07060302u);
            o[2][q] = __builtin_amdgcn_perm(t3, t1, 0x05040100u);
            o[3][q] = __builtin_amdgcn_perm(t3, t1, 0x07060302u);
        }
        gchar* dst = myws + slot + (size_t)sg * GS;
#pragma unroll
        for (int j = 0; j < 4; ++j)
            store16_sys(dst + (xq == 7 ? (28 + (cg >> 2)) * 1024 + (r - 28) * 256 + j * 64 + (cg & 3) * 16
                                       : r * 1024 + j * 256 + (cg ^ (4 * j)) * 16), o[j]);
        wait_vm<0>();
        gu32* const ac = (gu32*)(myws + AC_OFF + gp * 128);
        if (lds_ticket(CTL + 4, lane) == 1u) {
            if (xq) {
                if (lane == 0) __hip_atomic_fetch_add(ac, 1u, __ATOMIC_RELAXED, __HIP_MEMORY_SCOPE_AGENT);
            } else {
                gchar* peer_l = (gchar*)p.ws[lane & 7];
                unsigned tx;
                while ((tx = __builtin_amdgcn_readfirstlane(lds_ld(CTL + 12))) == 0u) __builtin_amdgcn_s_sleep(1);
                wait_until(tx - 1u, TF);
                if (lane < NR)
                    __hip_atomic_store((gu32*)(peer_l + FW_OFF + (gp * 8 + rank) * 128), call,
                                       __ATOMIC_RELAXED, __HIP_MEMORY_SCOPE_SYSTEM);
                ok = wait_flags<2>(ac, call * 7u, call, tmo);
                if (ok && lane < NR)
                    __hip_atomic_store((gu32*)(peer_l + FA_OFF + gp * 128) + rank, call,
                                       __ATOMIC_RELAXED, __HIP_MEMORY_SCOPE_SYSTEM);
            }
        }
        PUT_CALL();
    } else if (wave > 4) {
        GET_CALL();
        PUT_CALL();
        if (wave == MW) {
            while (__builtin_amdgcn_readfirstlane(lds_ld(CTL + 20)) == 0u) __builtin_amdgcn_s_sleep(4);
            __builtin_amdgcn_s_sleep(32);
            ok = wait_flags<16>((const gu32*)(myws + FA_OFF + (g >> 1) * 128) + (lane & 7), call, call, tmo);
            lds_st(CTL + 16, ok ? 1u : 2u);
        }
    } else {
        __builtin_amdgcn_s_setprio(3);
        const unsigned xcc = (unsigned)__builtin_amdgcn_s_getreg((3 << 11) | 20) & 0xFu;
        gu32* const iv = (gu32*)(myws + INV_OFF + xcc * 256);
        bool inv_done = false;
        if (b < 8) {
            asm volatile("buffer_inv sc0 sc1\n\ts_waitcnt vmcnt(0)" ::: "memory");
            inv_done = true;
        }
        GET_CALL();
        if (b < 8 && lane == 0) __hip_atomic_store(iv, call, __ATOMIC_RELAXED, __HIP_MEMORY_SCOPE_AGENT);
        PUT_CALL();
        const gchar* base[NR];
        unsigned ldst[NR];
#pragma unroll
        for (int i = 0; i < NR; ++i) {
            const unsigned rel = i < NR - 1 ? 1u + (unsigned)(g + i) % (NR - 1) : 0u;
            base[i] = (const gchar*)p.ws[rel] + slot;
            ldst[i] = lds0 + ((rel + rank) & 7u) * 1024;
        }
        {
            const gu32* fw = lane < NR ? (const gu32*)(myws + FW_OFF + ((g >> 1) * 8 + lane) * 128) : iv;
            bool seen = lane > NR || (lane == NR && inv_done);
            wait_until(t_start, PT0);
            const unsigned long long t0 = __builtin_amdgcn_s_memrealtime();
            for (unsigned spin = 1;; ++spin) {
                if (!seen) seen = __hip_atomic_load(fw, __ATOMIC_RELAXED, __HIP_MEMORY_SCOPE_SYSTEM) >= call;
                const unsigned long long rdy = __ballot(seen);
                inv_done = inv_done || ((rdy >> NR) & 1);
                if ((rdy & 0xFFull) == 0xFFull && inv_done) break;
                if ((spin & 31u) == 0u) {
                    if (!inv_done) {
                        asm volatile("buffer_inv sc0 sc1\n\ts_waitcnt vmcnt(0)" ::: "memory");
                        inv_done = true;
                    }
                    if (give_up(tmo, call, t0)) { ok = false; break; }
                }
                __builtin_amdgcn_s_sleep(1);
            }
        }
        const unsigned tref = (unsigned)__builtin_amdgcn_s_memrealtime();
        lds_st(CTL + 20, 1u);
        unsigned voff = g * GS + lane * 16;
#pragma unroll 1
        for (int r = 0; r < ROUNDS && ok; ++r) {
            if (r == 4) {
                unsigned fa;
                while ((fa = __builtin_amdgcn_readfirstlane(lds_ld(CTL + 16))) == 0u) __builtin_amdgcn_s_sleep(1);
                if (fa != 1u) { ok = false; break; }
            }
            const unsigned sched = PACE * (unsigned)r;
            const unsigned now = (unsigned)__builtin_amdgcn_s_memrealtime() - tref;
            const bool behind = now > sched + LAG;
            if (now < sched) wait_until(tref, sched);
            if (r >= RING) {
                for (;;) {
                    const unsigned dn = lds_ld(CTL + 32 + (lane & (MW - 1)) * 4);
                    if (__all(dn + RING > (unsigned)r)) break;
                    wait_vm<0>();
                    lds_st(CTL, (unsigned)r);
                    __builtin_amdgcn_s_sleep(1);
                }
            }
            const unsigned so = (unsigned)(r % RING) * 8192;
            if (behind) {
#pragma unroll
                for (int i = 0; i < NR; ++i) {
                    glds16(base[i], voff, ldst[i] + so);
                    wait_vm<WIN - 1 + CATCH>();
                }
                wait_vm<NR>();
                lds_st(CTL, (unsigned)r);
            } else {
#pragma unroll
                for (int i = 0; i < NR; ++i) {
                    glds16(base[i], voff, ldst[i] + so);
                    wait_vm<WIN - 1>();
                    if (i == NR - 2) lds_st(CTL, (unsigned)r);
                }
            }
            voff += 1024;
        }
        wait_vm<0>();
        lds_st(CTL, ok ? (unsigned)ROUNDS : 0x80000000u);
        return;
    }

    unsigned landed = 0;
    while ((landed = __builtin_amdgcn_readfirstlane(lds_ld(CTL))) < 2u) __builtin_amdgcn_s_sleep(8);
    const int mw = wave > 4 ? wave - 1 : wave;
    const int rt = mt * 8 + mw;
    ok = wait_flags<16>((const gu32*)(myws + FX_OFF + (rt >> 1) * 128) + (rt & 1) * 8 + (lane & 7), call, call, tmo) && ok;
    const gchar* abase = myws + slot + XST_OFF + (size_t)(rt * NR + (lane >> 4) * 2) * XSS + (lane & 15) * 16;
    constexpr int PF = 4;
    i32x4 a[PF][2];
#pragma unroll
    for (int u = 0; u < PF; ++u)
#pragma unroll
        for (int h = 0; h < 2; ++h) a[u][h] = *(const GAS i32x4*)(abase + h * XSS + u * 256);
    float sc = p.sx[0] * p.sw[0];
    float sce = -1.44269504088896341f * sc;
    GAS float* orow = (GAS float*)p.out + (size_t)(rt * 16 + (lane >> 2)) * NN + g * 64 + (lane & 3) * 4;
    const int from = ((lane & 3) * 16 + (lane >> 2)) * 4;
    asm volatile("" : "+v"(sc), "+v"(sce), "+v"(orow));
    constexpr float LIM = 60.f * 1.44269504088896341f;
    f32x4 acc[4];
#pragma unroll
    for (int j = 0; j < 4; ++j) acc[j] = f32x4{0.f, 0.f, 0.f, 0.f};
    const unsigned bl = (lane >> 4) * 2048 + (lane & 3) * 64 + ((lane >> 2) & 3) * 16;
    auto fetch = [&](int r, i32x8 (&bf)[4], bool tail) {
        while (landed <= (unsigned)r) {
            landed = __builtin_amdgcn_readfirstlane(lds_ld(CTL));
            if (landed <= (unsigned)r) __builtin_amdgcn_s_sleep(1);
        }
        const unsigned char* bs = smem + (r % RING) * 8192;
#pragma unroll
        for (int j = 0; j < 4; ++j) {
            const unsigned off = tail ? bl + j * 256 : ((bl + (lane & 3) * 256) ^ (j * 64));
            const i32x4 lo = *(const i32x4*)(bs + off);
            const i32x4 hi = *(const i32x4*)(bs + off + 1024);
            bf[j] = __builtin_shufflevector(lo, hi, 0, 1, 2, 3, 4, 5, 6, 7);
        }
    };
#pragma unroll 1
    for (int r0 = 0; r0 < ROUNDS - PF; r0 += PF) {
#pragma unroll
        for (int u = 0; u < PF; ++u) {
            const int r = r0 + u;
            i32x8 bf[4];
            fetch(r, bf, false);
            lds_st(CTL + 32 + mw * 4, (unsigned)r + 1u);
            __builtin_amdgcn_sched_barrier(0);
            const i32x8 af = __builtin_shufflevector(a[u][0], a[u][1], 0, 1, 2, 3, 4, 5, 6, 7);
#pragma unroll
            for (int j = 0; j < 4; ++j)
                acc[j] = __builtin_amdgcn_mfma_scale_f32_16x16x128_f8f6f4(bf[j], af, acc[j], 0, 0,
                                                                          0, 0x7F7F7F7F, 0, 0x7F7F7F7F);
            asm volatile("" : "+v"(acc[0]), "+v"(acc[1]), "+v"(acc[2]), "+v"(acc[3]));
            if (r0 >= PRE0 && r0 < PRE0 + 16 && u == (mw & 3)) {
                const f32x4 z = {__builtin_nanf(""), __builtin_nanf(""), __builtin_nanf(""), __builtin_nanf("")};
                asm volatile("global_store_dwordx4 %0, %1, off sc1\n\ts_nop 1"
                             :: "v"(orow + (r0 - PRE0) / PF * 16), "v"(z) : "memory");
            }
#pragma unroll
            for (int h = 0; h < 2; ++h) a[u][h] = *(const GAS i32x4*)(abase + h * XSS + (r + PF) * 256);
        }
    }
    wait_vm<0>();
#pragma unroll
    for (int u = 0; u < 4; ++u) {
        i32x8 bf[4];
        fetch(ROUNDS - 4 + u, bf, true);
        __builtin_amdgcn_sched_barrier(0);
#pragma unroll
        for (int rr = 0; rr < 4; ++rr) {
            const i32x8 af = __builtin_shufflevector(a[rr][0], a[rr][1], 0, 1, 2, 3, 4, 5, 6, 7);
            acc[u] = __builtin_amdgcn_mfma_scale_f32_16x16x128_f8f6f4(bf[rr], af, acc[u], 0, 0,
                                                                      0, 0x7F7F7F7F, 0, 0x7F7F7F7F);
        }
        if (!ok || landed > (unsigned)ROUNDS) sc = __builtin_nanf("");
        f32x4 o;
#pragma unroll
        for (int i = 0; i < 4; ++i) {
            const float e = __builtin_amdgcn_exp2f(__builtin_amdgcn_fmed3f(acc[u][i] * sce, -LIM, LIM));
            const float y = (acc[u][i] * sc) * __builtin_amdgcn_rcpf(1.f + e);
            o[i] = __int_as_float(__builtin_amdgcn_ds_bpermute(from, __float_as_int(y)));
        }
        asm volatile("global_store_dwordx4 %0, %1, off sc1\n\ts_nop 1" :: "v"(orow + u * 16), "v"(o) : "memory");
    }
}

}

extern "C" void kernel_launch(int rank, int nranks, ncclComm_t comm, hipStream_t stream,
                              void* const* d_in, const int* in_sizes, int n_in,
                              void* d_out, int out_size, const int* in_dtypes, int out_dtype,
                              void* const* d_out_all, void* const* d_ws_all, size_t ws_bytes) {
    (void)comm; (void)in_dtypes; (void)out_dtype; (void)d_out_all;
    if (nranks != NR || (unsigned)rank >= (unsigned)NR || ws_bytes < WS_NEED || n_in != 4
        || in_sizes[0] != NR * MS * KS || in_sizes[1] != KS * NN || out_size != MS * NN) {
        fprintf(stderr, "unexpected configuration\n");
        return;
    }
    Params p;
    p.x = (const unsigned char*)d_in[0];
    p.w = (const unsigned char*)d_in[1];
    p.sx = (const float*)d_in[2];
    p.sw = (const float*)d_in[3];
    p.out = (float*)d_out;
    for (int i = 0; i < NR; ++i) p.ws[i] = (unsigned char*)d_ws_all[(rank + i) % NR];
    p.myws = p.ws[0];
    p.rank = rank;
    p.pad = 0;
    p.pad2 = nullptr;
    gemm_gather_silu<<<NWG, (MW + 1) * 64, 0, stream>>>(p);
    HIP_CHECK(hipGetLastError());
}
```
